```python
import numpy as np

kernel_name = "gpu_gemm_a2a_m8192_k8192_n8192_bf16_gelu"


ABSMAX_RTOL = 0.008

COMPUTE_DTYPE = "bfloat16"
EPILOGUE = "gelu"
IN_QUANT = None
OUT_QUANT = None
IN_DTYPES = {"x": "bfloat16", "w_mat": "bfloat16"}
WORLD_SIZE = 8

def _to_bf16(x):
    u = np.ascontiguousarray(x, dtype=np.float32).view(np.uint32)
    return ((u + 0x7FFF + ((u >> 16) & 1)) & 0xFFFF0000).view(np.float32)

def setup_inputs():

    x = _to_bf16(np.random.randn(8192, 8192).astype(np.float32))
    w_mat = _to_bf16((np.random.randn(8192, 8192) / np.sqrt(8192)).astype(np.float32))
    return {"x": x, "w_mat": w_mat}

def _epilogue(y):
    c = 0.7978845608028654
    return (0.5 * y * (1.0 + np.tanh(c * (y + 0.044715 * y ** 3)))).astype(np.float32)

def reference(x, w_mat):
    y = (x @ w_mat).astype(np.float32)
    return _epilogue(y)
```

```cpp
#include <hip/hip_runtime.h>
#include <rccl/rccl.h>
#include <cstdio>

#define LAS __attribute__((address_space(3)))
typedef __bf16 bf16x4 __attribute__((ext_vector_type(4)));
typedef __bf16 bf16x8 __attribute__((ext_vector_type(8)));
typedef __bf16 bf16x2 __attribute__((ext_vector_type(2)));
typedef float f32x4 __attribute__((ext_vector_type(4)));
typedef float f32x2 __attribute__((ext_vector_type(2)));
typedef unsigned u32x4 __attribute__((ext_vector_type(4)));
typedef unsigned u32x2 __attribute__((ext_vector_type(2)));

#define BETA 1
#define DELTA 3
#define ETW 0
#define XPF 0
#define XDEL 8
#define XST 1
#define XFI 2
#define XSP 2
#define ELOCK 0
#define EB2 0
#define EPH 2
#define FPOS e
#define NL "\n\t"
#define ABL_F 4
#define ABL_S 1
#define ABL_A 17
#define ABL_D 1
#define SKIP(g, l, i) ((g) < 2 ? ((0x4a5 >> (4 * ((i) % 3))) & 15) : (l) ? 0 : (1 << ((i) & 3)))
#define SLM 1023
#define PINGPONG 1
#define AUXB 0
#define ABL 0
#define BLAST 1
#define PLAN 2
#define SMASK(g, ltype) (((g) == 2 && (ltype)) ? 0xa : 0xd)
constexpr int NR = 8;
constexpr int MS = 1024, KD = 8192, ND = 8192, NS_OUT = 1024;
constexpr int BM = 256, BN = 128, BK = 64;
constexpr int A_BYTES = BM * BK * 2, B_BYTES = BK * BN * 2, STAGE_BYTES = A_BYTES + B_BYTES;
constexpr int RING_BYTES = 3 * STAGE_BYTES;
constexpr int LDS_BYTES = RING_BYTES + 2 * 8192;
constexpr unsigned long long DEADLINE = 50000000ull;
constexpr int AUX_SYS = 17;
static_assert((BETA + DELTA) % 2 == 0 && BETA + DELTA <= 24, "generation 0 must keep an even, positive length");

struct Params { const void* x; const void* w; void* out[NR]; unsigned* ws[NR]; int rank; unsigned tab[256]; };
constexpr int WS_TMO = 0;
constexpr int WS_CNT = 32;
constexpr int WS_DUMP = WS_CNT + 256 * 32;
constexpr int WS_DUMP3 = WS_DUMP + 3 * 256 * 32;
constexpr int WS_RECV = WS_DUMP3 + 1024 * 32;
constexpr int WS_END = WS_RECV + 2560 * 32;
constexpr size_t PART_OFF = 1u << 20;
constexpr unsigned SLAB = 65536u;
constexpr unsigned PART_BYTES = 4u * 256u * SLAB;
static_assert(WS_END <= 163840, "flags area");

__device__ __forceinline__ float gelu1(float y) {
  const float k1 = -2.0f * 0.7978845608028654f * 1.4426950408889634f;
  const float k2 = k1 * 0.044715f;
  float t = y * y;
  float u = __builtin_fmaf(t, k2, k1) * y;
  float e = __builtin_amdgcn_exp2f(u);
  return y * __builtin_amdgcn_rcpf(1.0f + e);
}
__device__ __forceinline__ unsigned pk2(float a, float b) {
  f32x2 v = {a, b}; bf16x2 r = __builtin_convertvector(v, bf16x2); return __builtin_bit_cast(unsigned, r);
}
__device__ __forceinline__ float e_add(float a, float b) { float r; asm volatile("v_add_f32 %0, %1, %2" NL : "=v"(r) : "v"(a), "v"(b)); return r; }
__device__ __forceinline__ float e_lo(unsigned a) { float r; asm volatile("v_lshlrev_b32 %0, 16, %1" NL : "=v"(r) : "v"(a)); return r; }
__device__ __forceinline__ float e_hi(unsigned a) { float r; asm volatile("v_and_b32 %0, 0xffff0000, %1" NL : "=v"(r) : "v"(a)); return r; }
__device__ __forceinline__ unsigned e_gelu2(float y0, float y1, float k1v) {
  const float k2 = -2.0f * 0.7978845608028654f * 1.4426950408889634f * 0.044715f;
  float t0, t1; unsigned r;
  asm volatile("v_mul_f32 %1, %3, %3" NL "v_mul_f32 %2, %4, %4" NL "v_fma_f32 %1, %1, %5, %6" NL "v_fma_f32 %2, %2, %5, %6" NL "v_mul_f32 %1, %1, %3" NL "v_mul_f32 %2, %2, %4" NL
               "v_exp_f32 %1, %1" NL "v_exp_f32 %2, %2" NL "v_add_f32 %1, 1.0, %1" NL "v_add_f32 %2, 1.0, %2" NL "v_rcp_f32 %1, %1" NL "v_rcp_f32 %2, %2" NL
               "v_mul_f32 %1, %1, %3" NL "v_mul_f32 %2, %2, %4" NL "v_cvt_pk_bf16_f32 %0, %1, %2" NL : "=v"(r), "=&v"(t0), "=&v"(t1) : "v"(y0), "v"(y1), "s"(k2), "v"(k1v));
  return r;
}
#define BAR() asm volatile("s_barrier" ::: "memory")
#define WAITV(n) asm volatile("s_waitcnt vmcnt(" #n ")" ::: "memory")
#define SCHED0() __builtin_amdgcn_sched_barrier(0)
#define RFL(x) __builtin_amdgcn_readfirstlane(x)
#define PIN(x) asm volatile("" : "+v"(x))

#define STAMPAT(i)
#define STAMPOUT(base)

__device__ __forceinline__ bool wait_flag(const unsigned* flag, unsigned call, unsigned* tmo) {
  const unsigned long long t0 = __builtin_amdgcn_s_memrealtime();
  bool bad = false;
  while (__hip_atomic_load(flag, __ATOMIC_RELAXED, __HIP_MEMORY_SCOPE_SYSTEM) < call) {
    if (__hip_atomic_load(tmo, __ATOMIC_RELAXED, __HIP_MEMORY_SCOPE_AGENT) == call || __builtin_amdgcn_s_memrealtime() - t0 > DEADLINE) {
      __hip_atomic_store(tmo, call, __ATOMIC_RELAXED, __HIP_MEMORY_SCOPE_AGENT);
      bad = true; break;
    }
    __builtin_amdgcn_s_sleep(1);
  }
  return bad;
}

__global__ __launch_bounds__(512, 2) void gemm_a2a(Params p) {
  __shared__ __attribute__((aligned(1024))) char smem[LDS_BYTES];
  LAS char* lds = (LAS char*)smem;
  const unsigned ldsb = (unsigned)(size_t)lds;
  const int tid = threadIdx.x;
  const int wid = RFL(tid >> 6);
  const int lane = tid & 63;
  const int b = blockIdx.x;
  const int xj = b & 7, idx = b >> 3, q = xj & 3;
  const int vb = b ^ ((xj & 1) << 7);
  const bool ltype = vb >= 128;
  const unsigned tab = p.tab[b];
  unsigned* const myws = p.ws[p.rank];
  const int cx = 6 ^ (2 * q);
  const int G0 = 32 + ((xj & 1) ? -BETA : BETA) + (ltype ? DELTA : -DELTA);
  const int NST = G0 + 96;

  if (wid >= 6) {
    const int e = wid - 6;
    unsigned call = RFL(__hip_atomic_load(myws + WS_CNT + b * 32, __ATOMIC_RELAXED, __HIP_MEMORY_SCOPE_AGENT)) + 1;
    int nb = NST + 2;
#define STEP() do { if (nb > 0) { BAR(); --nb; } SCHED0(); } while (0)
#define EMARK(k)
#define ERESET()
    auto rp = __builtin_amdgcn_make_buffer_rsrc((char*)myws + PART_OFF, 0, PART_BYTES, 0x00020000);
    const unsigned vld = (unsigned)((lane >> 5) * 1024 + ((lane >> 3) & 3) * 256 + (lane & 7) * 16);
    const unsigned vrd = ldsb + RING_BYTES + e * 8192 + (unsigned)(((lane >> 3) & 1) * 4096 + (((lane >> 2) & 1) * 4 + (lane & 1) * 2) * 128 + (lane >> 4) * 16 + ((lane >> 1) & 1) * 8);
    const unsigned vst = (unsigned)((lane >> 4) * (NS_OUT * 2) + (lane & 15) * 16);
    bool bad = false;
    float k1v = -2.0f * 0.7978845608028654f * 1.4426950408889634f; PIN(k1v);
    for (int g = 0; g < 3; ++g) {
      const int s0 = g * 256 + (g == 0 ? b : vb) - q;
      const unsigned te = (tab >> (8 * g)) & 0xff;
      auto ro = __builtin_amdgcn_make_buffer_rsrc(p.out[(te >> 3) & 7], 0, 8192 * NS_OUT * 2, 0x00020000);
      const int lag = g == 0 && G0 < 32 ? 64 - 2 * G0 : 0;
      while (nb > 96 - 32 * g - 4 - lag) STEP();
      {
        const unsigned long long t0 = __builtin_amdgcn_s_memrealtime();
        const unsigned* fl = myws + WS_DUMP + (s0 + (lane & 3)) * 32;
        for (;;) {
          unsigned v, tv;
          asm volatile("global_load_dword %0, %2, off sc0 sc1\n\tglobal_load_dword %1, %3, off sc1" : "=&v"(v), "=&v"(tv) : "v"(fl), "v"(myws + WS_TMO) : "memory");
          STEP();
          asm volatile("s_waitcnt vmcnt(0)" : "+v"(v), "+v"(tv) :: "memory");
          if (__all(v >= call)) break;
          if (RFL(tv) == call || __builtin_amdgcn_s_memrealtime() - t0 > DEADLINE) {
            if (lane == 0) __hip_atomic_store(myws + WS_TMO, call, __ATOMIC_RELAXED, __HIP_MEMORY_SCOPE_AGENT);
            bad = true; break;
          }
          if (nb == 0) __builtin_amdgcn_s_sleep(1);
        }
      }
      auto fetch = [&](int u) {
        const int sl = q * 16 + u * 4 + e * 2;
        const unsigned sld = (unsigned)((s0 & SLM) * SLAB + (sl >> 5) * 32768 + ((sl >> 2) & 7) * 2048 + (sl & 3) * 64);
#pragma unroll
        for (int s = 0; s < 8; ++s) if (!(ABL & 1) || (s & 3) != q) if (!(ABL & 2) || g > 0) if (!(ABL & 8))
          __builtin_amdgcn_raw_ptr_buffer_load_lds(rp, (LAS void*)(lds + RING_BYTES + e * 8192 + s * 1024), 16, vld, sld + (s & 3) * SLAB + (s >> 2) * 16384, 0, AUX_SYS);
      };
      const int skip = SKIP(g, ltype, idx);
      fetch(0);
      STEP();
      while (nb > 0 && ((NST + 2 - nb + ((idx >> 2) & 3) + ELOCK) & 3)) STEP();
      for (int u = 0; u < 4; ++u) {
        if ((skip >> u) & 1) { STEP(); STEP(); STEP(); STEP(); }
        if (u > 0) WAITV(2); else WAITV(0);
        u32x2 L[2][4][2]; u32x4 o[2];
#pragma unroll
        for (int h = 0; h < 2; ++h)
          asm volatile("ds_read_b64 %0, %8\n\tds_read_b64 %1, %8 offset:128\n\tds_read_b64 %2, %8 offset:1024\n\tds_read_b64 %3, %8 offset:1152\n\t"
                       "ds_read_b64 %4, %8 offset:2048\n\tds_read_b64 %5, %8 offset:2176\n\tds_read_b64 %6, %8 offset:3072\n\tds_read_b64 %7, %8 offset:3200\n\ts_waitcnt lgkmcnt(0)"
                       : "=&v"(L[h][0][0]), "=&v"(L[h][0][1]), "=&v"(L[h][1][0]), "=&v"(L[h][1][1]), "=&v"(L[h][2][0]), "=&v"(L[h][2][1]), "=&v"(L[h][3][0]), "=&v"(L[h][3][1])
                       : "v"(vrd + h * 64) : "memory");
        SCHED0();
        float y[8];
        auto sum = [&](int h, int d) {
          float y0 = e_lo(L[h][0][d >> 1][d & 1]), y1 = e_hi(L[h][0][d >> 1][d & 1]);
#pragma unroll
          for (int s = 1; s < 4; ++s) { const float a0 = e_lo(L[h][s][d >> 1][d & 1]), a1 = e_hi(L[h][s][d >> 1][d & 1]); y0 = e_add(y0, a0); y1 = e_add(y1, a1); }
          y[2 * d] = y0; y[2 * d + 1] = y1;
        };
        auto act = [&](int h) {
#pragma unroll
          for (int d = 0; d < 4; ++d) o[h][d] = e_gelu2(y[2 * d], y[2 * d + 1], k1v);
          if (bad) o[h] = (u32x4){0x7fc07fc0u, 0x7fc07fc0u, 0x7fc07fc0u, 0x7fc07fc0u};
        };
        sum(0, 0); sum(0, 1); sum(0, 2); sum(0, 3);
        STEP();
        act(0);
        SCHED0();
        if (e == 0 && u < 3) fetch(u + 1);
        STEP();
        sum(1, 0); sum(1, 1); sum(1, 2); sum(1, 3);
        SCHED0();
        if (e == 1 && u < 3) fetch(u + 1);
        STEP();
        act(1);
        const unsigned so = (unsigned)((p.rank * MS + (te >> 6) * BM + (q * 16 + u * 4 + e * 2) * 4) * (NS_OUT * 2) + (te & 7) * (BN * 2));
        __builtin_amdgcn_raw_buffer_store_b128(o[0], ro, vst, so, AUX_SYS);
        __builtin_amdgcn_raw_buffer_store_b128(o[1], ro, vst, so + 4 * (NS_OUT * 2), AUX_SYS);
        asm volatile("s_nop 1");
        STEP();
      }
    }
    STEP();
    WAITV(0);
    STAMPAT(4);
    const unsigned dl = (tab >> (8 * (lane & 3) + 3)) & 7;
    const int slot = (b * 10 + lane * 2 + e) * 32;
    if (lane < 3 && dl != (unsigned)p.rank) __hip_atomic_store(p.ws[dl] + WS_RECV + slot, call, __ATOMIC_RELAXED, __HIP_MEMORY_SCOPE_SYSTEM);
    while (nb > 0) STEP();
    STAMPAT(5);
    if (lane < 3) {
      if (dl != (unsigned)p.rank) {
        if (wait_flag(myws + WS_RECV + slot, call, myws + WS_TMO))
          __hip_atomic_store((unsigned short*)p.out[p.rank] + (size_t)(p.rank * MS) * NS_OUT, (unsigned short)0x7fc0, __ATOMIC_RELAXED, __HIP_MEMORY_SCOPE_SYSTEM);
      }
    }
    if (e == 0 && lane == 0) __hip_atomic_store(myws + WS_CNT + b * 32, call, __ATOMIC_RELAXED, __HIP_MEMORY_SCOPE_AGENT);
    STAMPAT(6);
    if (e == 0) STAMPOUT(180224);
  } else if (wid >= 4) {
    const int dw = wid - 4;
    unsigned vcall;
    asm volatile("global_load_dword %0, %1, off sc1" : "=v"(vcall) : "v"(myws + WS_CNT + b * 32) : "memory");
    constexpr int NA = 16, NB = 8;
    auto rx = __builtin_amdgcn_make_buffer_rsrc((char*)p.x - 4096, 0, MS * KD * 2 + 4096, 0x00020000);
    auto rw = __builtin_amdgcn_make_buffer_rsrc((void*)p.w, 0, KD * ND * 2, 0x00020000);
    unsigned vA[NA], vB[NB];
#pragma unroll
    for (int j = 0; j < NB; ++j) { vB[j] = (unsigned)((4 * (dw * NB + j) + (lane >> 4)) * (ND * 2) + (((lane & 15) ^ (((lane >> 4) | (((j >> 1) & 1) << 2)) << 1)) * 16) - (j & 3) * 1024); PIN(vB[j]); }
#pragma unroll
    for (int i = 0; i < NA; ++i) { vA[i] = (unsigned)((8 * ((dw * NA + i) ^ (2 * cx)) + (lane >> 3)) * (KD * 2) + 4096 + (((lane & 7) ^ ((4 * (i & 1) + (lane >> 4)) & 7)) * 16) - (i & 3) * 1024); PIN(vA[i]); }
    auto calcB = [&](int t) -> unsigned {
      t = t < NST ? t : NST - 1;
      const int g = t < G0 ? 0 : ((t - G0) >> 5) + 1;
      const unsigned te = tab >> (8 * g);
      const int ph = g == 0 && xj < 4 && idx < 12 ? (int)((te >> 3) & 3) : (idx >> 2) & 3, lo = g == 0 ? 0 : G0 + 32 * (g - 1), hi = G0 + 32 * g, bs = ((t + ph) & ~3) - ph;
      if (bs >= lo && bs + 4 <= hi) t = ((t + ph) ^ ((te >> 6) & 3)) - ph;
      const int r = (xj >> 2) * 32;
      const int k0 = (xj & 2) * 32 + (((xj & 1) ? r - 1 - t : r + t) & 63);
      const int u = t - lo, up = (g ^ xj) & 1 ? 0 : 1;
      const int k1 = PINGPONG ? (xj & 2) * 32 + ((r + (xj & 1) * 32 + (up ? u : 31 - u)) & 63) : q * 32 + ((t + (xj >> 2) * 16) & 31);
      return (te & 63) * (BN * 2) + (unsigned)(g == 0 ? k0 : k1) * (BK * ND * 2) + ((te & 0xc0) << 22);
    };
    const unsigned vt0 = calcB(lane), vt1 = calcB(64 + lane), vt2 = calcB(128 + lane);
    auto offB = [&](int t) -> unsigned {
      const unsigned u0 = __builtin_amdgcn_readlane(vt0, t & 63), u1 = __builtin_amdgcn_readlane(vt1, t & 63), u2 = __builtin_amdgcn_readlane(vt2, t & 63);
      return t < 64 ? u0 : t < 128 ? u1 : u2;
    };
    auto b2a = [&](unsigned sB) -> unsigned { return (sB >> 28) * (BM * KD * 2) + ((sB >> 20) & 127) * (BK * 2); };
#define Q4(r, l, v, i, s, X) do { __builtin_amdgcn_raw_ptr_buffer_load_lds(r, (LAS void*)(l), 16, v[i], s, 0, X); __builtin_amdgcn_raw_ptr_buffer_load_lds(r, (LAS void*)(l), 16, v[i + 1], s, 1024, X); \
  __builtin_amdgcn_raw_ptr_buffer_load_lds(r, (LAS void*)(l), 16, v[i + 2], s, 2048, X); __builtin_amdgcn_raw_ptr_buffer_load_lds(r, (LAS void*)(l), 16, v[i + 3], s, 3072, X); } while (0)
    auto issueB = [&](unsigned sB, unsigned lb) {
      sB &= 0x0fffffffu;
#pragma unroll
      for (int j = 0; j < NB; j += 4) Q4(rw, (size_t)(lb + A_BYTES + (dw * NB + j) * 1024), vB, j, sB, AUXB);
    };
    auto issueA = [&](unsigned sA, unsigned lb) {
#pragma unroll
      for (int i = 0; i < NA; i += 4) Q4(rx, (size_t)(lb + (dw * NA + i) * 1024), vA, i, sA, 0);
    };
    issueB(offB(0), ldsb + 2 * STAGE_BYTES); issueA(b2a(offB(0)), ldsb); issueB(offB(1), ldsb); issueA(b2a(offB(1)), ldsb + STAGE_BYTES); issueB(offB(2), ldsb + STAGE_BYTES);
    asm volatile("s_waitcnt vmcnt(24)" : "+v"(vcall) :: "memory");
    BAR();
    vcall += 1;
    BAR();
    issueA(b2a(offB(2)), ldsb + 2 * STAGE_BYTES); issueB(offB(3), ldsb + 2 * STAGE_BYTES);
    unsigned lb = ldsb;
    unsigned sB = offB(4), sA = b2a(offB(3));
    int pub = G0 + 2;
    for (int s = 0; s < NST - 3; ++s) {
      WAITV(24); BAR();
      if (s == pub) {
        if (dw == 0 && lane == 0)
          __hip_atomic_store(myws + WS_DUMP + (s == G0 + 2 ? b : (s - G0 - 2) * 8 + vb) * 32, vcall, __ATOMIC_RELAXED, __HIP_MEMORY_SCOPE_SYSTEM);
        pub += 32;
      }
      if (BLAST) issueA(sA, lb); else issueB(sB, lb);
      SCHED0();
      unsigned nA = b2a(sB), nB = offB(s + 5);
      asm volatile("" : "+s"(nB), "+s"(nA));
      SCHED0();
      if (BLAST) issueB(sB, lb); else issueA(sA, lb);
      sA = nA; sB = nB;
      lb = lb == ldsb + 2 * STAGE_BYTES ? ldsb : lb + STAGE_BYTES;
    }
    WAITV(24); BAR();
    WAITV(0); BAR();
    BAR();
  } else {
    unsigned vcall;
    asm volatile("global_load_dword %0, %1, off sc1" : "=v"(vcall) : "v"(myws + WS_CNT + b * 32) : "memory");
    const int wm = wid >> 1, wn = wid & 1;
    unsigned aoff[2], boff[4];
    {
      const int l15 = lane & 15, g = lane >> 4;
#pragma unroll
      for (int ks = 0; ks < 2; ++ks) aoff[ks] = ldsb + (unsigned)((wm * 128 + l15) * 128 + (((ks * 4 + g) ^ (l15 >> 1)) * 16));
      const int qq = l15 >> 2, pp = lane & 3, h = qq | ((g & 1) << 2);
#pragma unroll
      for (int ni = 0; ni < 4; ++ni) boff[ni] = ldsb + (unsigned)(2 * STAGE_BYTES + A_BYTES + (8 * g + qq) * 256 + (((wn * 4 + ni) ^ h) * 32) + 8 * pp);
    }
    f32x4 acc[8][4];
    bf16x8 a[4][2]; bf16x4 bb[2][4][2][2];
    int bufn = 0; unsigned dlt;

#define RDA(slot, ks, mi) a[slot][ks] = *(LAS bf16x8*)(size_t)(aoff[ks] + (mi) * 2048)
#define RDB(P, ni, ks, t) bb[P][ni][ks][t] = __builtin_amdgcn_ds_read_tr16_b64_v4bf16((LAS bf16x4*)(size_t)(boff[ni] + (ks) * 8192 + (t) * 1024))
#define BF(P, ni, ks) __builtin_shufflevector(bb[P][ni][ks][0], bb[P][ni][ks][1], 0, 1, 2, 3, 4, 5, 6, 7)
#define MF(ZERO, P, mi, ni, ks) do { if (ZERO) asm volatile("v_mfma_f32_16x16x32_bf16 %0, %1, %2, 0" : "+v"(acc[mi][ni]) : "v"(BF(P, ni, ks)), "v"(a[(mi) & 3][ks])); \
   else asm volatile("v_mfma_f32_16x16x32_bf16 %0, %1, %2, %0" : "+v"(acc[mi][ni]) : "v"(BF(P, ni, ks)), "v"(a[(mi) & 3][ks])); } while (0)
#define CVT(k, i, T) asm volatile("v_cvt_pk_bf16_f32 %0, %1, %2" : "=v"(T[i]) : "v"(acc[(k) >> 1][2 * ((k) & 1) + ((i) >> 1)][2 * ((i) & 1)]), "v"(acc[(k) >> 1][2 * ((k) & 1) + ((i) >> 1)][2 * ((i) & 1) + 1]))
#define PUT(k, T) if (!(ABL & 1) || (k) < 12) __builtin_amdgcn_raw_buffer_store_b128((u32x4){T[0], T[1], T[2], T[3]}, rd, vl + ((k) & 1) * 1024, sd + (unsigned)((((k) >> 1) ^ cx) * 2048), AUX_SYS)
#define STAGE(KIND, P, DRAIN) do { \
    _Pragma("unroll") for (int mi = 0; mi < 8; ++mi) { \
      _Pragma("unroll") for (int j = 0; j < 8; ++j) { \
        MF(KIND == 1 && (j & 1) == 0, P, mi, j >> 1, j & 1); SCHED0(); \
        if (j == 0) RDA((mi + 3) & 3, 0, (mi + 3) & 7); \
        if (j == (mi == 4 ? 2 : 4)) RDA((mi + 3) & 3, 1, (mi + 3) & 7); \
        if (mi < 4 && (j == 1 || j == 2 || j == 5 || j == 6)) { const int r_ = (j < 4 ? j - 1 : j - 3); RDB((P) ^ 1, mi, r_ >> 1, r_ & 1); } \
        if (mi == 3 && j == 3) { bufn = bufn == 2 ? 0 : bufn + 1; dlt = bufn == 0 ? (unsigned)(-2 * STAGE_BYTES) : (unsigned)STAGE_BYTES; asm volatile("" : "+s"(dlt), "+s"(bufn)); } \
        if (mi == 4 && j == 4) { aoff[0] += dlt; PIN(aoff[0]); } \
        if (mi == 4 && j == 5) { aoff[1] += dlt; PIN(aoff[1]); } \
        if (mi == 4 && j == 7) { if (DRAIN) WAITV(0); BAR(); } \
        if ((ABL & 4) && j == 7) asm volatile("s_nop 4"); \
        if (mi == 5 && j == 2) { boff[0] += dlt; PIN(boff[0]); } \
        if (mi == 5 && j == 6) { boff[1] += dlt; PIN(boff[1]); } \
        if (mi == 6 && j == 2) { boff[2] += dlt; PIN(boff[2]); } \
        if (mi == 6 && j == 6) { boff[3] += dlt; PIN(boff[3]); } \
        if (KIND == 2 && mi >= 1) { if (j < 4) CVT(mi - 1, j & 3, t0); if (j == 5) PUT(mi - 1, t0); } \
        if (KIND == 1) { \
          if (mi < 7) { if (j < 4) CVT(mi + 7, j & 3, t0); if (j == 5) PUT(mi + 7, t0); } \
          if (mi == 6 && j >= 4) CVT(14, j & 3, t1); \
          if (mi == 7) { if (j < 4) CVT(15, j & 3, t0); if (j == 1) PUT(14, t1); if (j == 6) PUT(15, t0); } \
        } \
        SCHED0(); \
      } \
    } } while (0)

    __builtin_amdgcn_s_setprio(2);
    STAMPAT(0);
    BAR();
    STAMPAT(1);
#pragma unroll
    for (int r = 0; r < 16; ++r) RDB(0, r >> 2, (r >> 1) & 1, r & 1);
#pragma unroll
    for (int mi = 0; mi < 3; ++mi) { RDA(mi, 0, mi); RDA(mi, 1, mi); }
#pragma unroll
    for (int ni = 0; ni < 4; ++ni) { boff[ni] -= 2 * STAGE_BYTES; PIN(boff[ni]); }
    asm volatile("s_waitcnt vmcnt(0)" : "+v"(vcall) :: "memory");
    const unsigned call = RFL(vcall) + 1;
    SCHED0();
    BAR();
    SCHED0();
    auto rp = __builtin_amdgcn_make_buffer_rsrc((char*)myws + PART_OFF, 0, PART_BYTES, 0x00020000);
    auto rd = __builtin_amdgcn_make_buffer_rsrc((char*)myws + PART_OFF, 0, 0, 0x00020000);
    unsigned vl = lane * 16; PIN(vl);
    unsigned sd = 0, t0[4], t1[4];
#pragma unroll
    for (int mi = 0; mi < 8; ++mi)
#pragma unroll
      for (int ni = 0; ni < 4; ++ni) asm volatile("" : "=v"(acc[mi][ni]));
    for (int g = 0; g < 4; ++g) {
      STAGE(1, 0, false);
      const int n = g == 0 ? G0 : 32;
      for (int s = 2; s < n; s += 2) {
        STAGE(0, 1, false);
        STAGE(0, 0, true);
      }
      if ((!(ABL & 2) || g > 0) && (!(ABL & 8) || g == 3)) rd = rp;
      sd = (unsigned)(((g * 256 + (g == 0 ? b : vb)) & SLM) * SLAB + wid * 16384);
      STAGE(2, 1, false);
    }
    asm volatile("s_nop 7");
    __builtin_amdgcn_s_setprio(0);
    STAMPAT(2);

    const int ln = __builtin_amdgcn_mbcnt_hi(-1, __builtin_amdgcn_mbcnt_lo(-1, 0));
    const int v0 = vb - q;
#pragma unroll
    for (int k = 7; k < 12; ++k) {
      unsigned t[4];
#pragma unroll
      for (int i = 0; i < 4; ++i) CVT(k, i, t);
      PUT(k, t); asm volatile("s_nop 1" :: "v"(t[0]), "v"(t[1]), "v"(t[2]), "v"(t[3]));
    }
    WAITV(0);
    STAMPAT(3);
    bool bad = false;
    if (ln == 0) __hip_atomic_store(myws + WS_DUMP3 + (vb * 4 + wid) * 32, call, __ATOMIC_RELAXED, __HIP_MEMORY_SCOPE_SYSTEM);
    if (ln < 4 && ln != q) bad = wait_flag(myws + WS_DUMP3 + ((v0 + ln) * 4 + wid) * 32, call, myws + WS_TMO);
    bad = __any(bad);
    asm volatile("" ::: "memory");
    STAMPAT(4);
    f32x4 own[2][4];
    {
      u32x4 L[3][2][2];
#pragma unroll
      for (int s = 0; s < 3; ++s) {
        const int pq = (q + 1 + s) & 3;
        const unsigned sl = (unsigned)(((3 * 256 + v0 + pq) & SLM) * SLAB + wid * 16384 + q * 4096);
#pragma unroll
        for (int m2 = 0; m2 < 2; ++m2)
#pragma unroll
          for (int h = 0; h < 2; ++h) L[s][m2][h] = __builtin_amdgcn_raw_buffer_load_b128(rp, ln * 16 + h * 1024, sl + m2 * 2048, AUX_SYS);
      }
#pragma unroll
      for (int m2 = 0; m2 < 2; ++m2)
#pragma unroll
        for (int ni = 0; ni < 4; ++ni) {
          own[m2][ni] = acc[6 + m2][ni];
#pragma unroll
          for (int s = 0; s < 3; ++s) {
            const unsigned u0 = L[s][m2][ni >> 1][(ni & 1) * 2], u1 = L[s][m2][ni >> 1][(ni & 1) * 2 + 1];
            own[m2][ni] += (f32x4){__builtin_bit_cast(float, u0 << 16), __builtin_bit_cast(float, u0 & 0xffff0000u), __builtin_bit_cast(float, u1 << 16), __builtin_bit_cast(float, u1 & 0xffff0000u)};
          }
        }
    }
    const unsigned te = tab >> 24;
    const int dest = (te >> 3) & 7;
    {
      LAS char* st = lds + wid * 4096;
      const int l15 = ln & 15, g = ln >> 4;
      const unsigned wo = (unsigned)(l15 * 128 + (g & 1) * 8);
#pragma unroll
      for (int m2 = 0; m2 < 2; ++m2)
#pragma unroll
        for (int ni = 0; ni < 4; ++ni) {
          f32x4 v = own[m2][ni];
          u32x2 o; o.x = pk2(gelu1(v[0]), gelu1(v[1])); o.y = pk2(gelu1(v[2]), gelu1(v[3]));
          if (bad) o = (u32x2){0x7fc07fc0u, 0x7fc07fc0u};
          *(LAS u32x2*)(st + m2 * 2048 + wo + (((ni * 2 + (g >> 1)) ^ (l15 & 7)) * 16)) = o;
        }
      __builtin_amdgcn_fence(__ATOMIC_RELEASE, "wavefront"); __builtin_amdgcn_wave_barrier(); __builtin_amdgcn_fence(__ATOMIC_ACQUIRE, "wavefront");
      auto ro = __builtin_amdgcn_make_buffer_rsrc(p.out[dest], 0, 8192 * NS_OUT * 2, 0x00020000);
      const unsigned vo = (unsigned)((ln >> 3) * (NS_OUT * 2) + (((ln & 7) ^ (ln >> 3)) * 16));
      const unsigned so = (unsigned)((p.rank * MS + (te >> 6) * BM + wm * 128 + q * 32) * (NS_OUT * 2) + (te & 7) * (BN * 2) + wn * 128);
#pragma unroll
      for (int j = 0; j < 4; ++j) {
        u32x4 r = *(LAS u32x4*)(st + j * 1024 + ln * 16);
        __builtin_amdgcn_raw_buffer_store_b128(r, ro, vo, so + j * 8 * (NS_OUT * 2), AUX_SYS);
        if (dest != p.rank) WAITV(0);
      }
    }
    STAMPAT(5);
    WAITV(0);
    STAMPAT(6);
    if (ln == 0 && dest != p.rank) {
      const int slot = (b * 10 + 6 + wid) * 32;
      __hip_atomic_store(p.ws[dest] + WS_RECV + slot, call, __ATOMIC_RELAXED, __HIP_MEMORY_SCOPE_SYSTEM);
      if (wait_flag(myws + WS_RECV + slot, call, myws + WS_TMO))
        __hip_atomic_store((unsigned short*)p.out[p.rank] + (size_t)(p.rank * MS) * NS_OUT, (unsigned short)0x7fc0, __ATOMIC_RELAXED, __HIP_MEMORY_SCOPE_SYSTEM);
    }
    STAMPAT(7);
    if (wid == 0) STAMPOUT(163840);
  }
}

extern "C" void kernel_launch(int rank, int nranks, ncclComm_t comm, hipStream_t stream,
                              void* const* d_in, const int* in_sizes, int n_in,
                              void* d_out, int out_size, const int* in_dtypes, int out_dtype,
                              void* const* d_out_all, void* const* d_ws_all, size_t ws_bytes) {
  if (nranks != NR || (unsigned)rank >= (unsigned)NR || ws_bytes < PART_OFF + (size_t)PART_BYTES) return;
  Params p;
  p.x = d_in[0]; p.w = d_in[1]; p.rank = rank;
  for (int i = 0; i < NR; ++i) { p.out[i] = d_out_all[i]; p.ws[i] = (unsigned*)d_ws_all[i]; }
  static const int label[8] = {0, 4, 1, 2, 5, 6, 3, 7};
  int peer[8]; for (int r = 0; r < 8; ++r) peer[label[rank] ^ label[r]] = r;
  unsigned char lst[8][32]; int cnt[8] = {0}, next[8] = {0};
  auto add = [&](int grp, int delta, int tn, int tm) { lst[grp][cnt[grp]++] = (unsigned char)(peer[delta] * 8 + tn + tm * 64); };
#define X 16
  static const unsigned char dl[64] = {8,8,8,1,2,3,1,2, 4,5,6,7,4,5,6,7,
                                       3+X,1+X,2+X,3,1,2,3,1, 4+X,5+X,6+X,7+X,4,5,6,7,
                                       2,3,1,2+X,3+X,1+X,2,3, 4,5,6,7,4+X,5+X,6+X,7+X,
                                       1,2,3,9,0,0,0+X,0+X, 4,5,6,7,0,0,0,0};
#undef X
  static const unsigned char good[6] = {0, 2, 3, 4, 6, 7};
  int nextb[8] = {0};
  bool far_done = false;
  for (int i = 0; i < 64; ++i) {
    const int d0 = dl[i] & 15;
    if (d0 == 8) {
      if (!far_done) for (int d = 4; d < 8; ++d) for (int tm = 0; tm < 4; ++tm) if (tm != d - 4) add(i >> 3, d, 7, tm);
      far_done = true;
    } else if (d0 == 9) {
      for (int d = 4; d < 8; ++d) add(i >> 3, d, 7, d - 4);
    } else {
      const int tn = PLAN == 1 ? next[d0]++ : (dl[i] & 16) ? 1 + 4 * nextb[d0]++ : good[next[d0]++];
      for (int tm = 0; tm < 4; ++tm) add(i >> 3, d0, tn, tm);
    }
  }
  for (int b = 0; b < 256; ++b) {
    const int xj = b & 7, i = b >> 3, vi = i ^ ((xj & 1) << 4), h = xj >> 2;
    p.tab[b] = lst[h][i] | (lst[2 + h][vi] << 8) | (lst[4 + h][vi] << 16) | ((unsigned)lst[6 + h][vi] << 24);
  }
  gemm_a2a<<<256, 512, 0, stream>>>(p);
}
```

```cpp
#include <hip/hip_runtime.h>
#include <rccl/rccl.h>
#include <cstdio>
#include <type_traits>

#define LAS __attribute__((address_space(3)))
typedef __bf16 bf16x4 __attribute__((ext_vector_type(4)));
typedef __bf16 bf16x8 __attribute__((ext_vector_type(8)));
typedef __bf16 bf16x2 __attribute__((ext_vector_type(2)));
typedef float f32x4 __attribute__((ext_vector_type(4)));
typedef float f32x2 __attribute__((ext_vector_type(2)));
typedef unsigned u32x4 __attribute__((ext_vector_type(4)));
typedef unsigned u32x2 __attribute__((ext_vector_type(2)));

#define BETA 1
#define DELTA 3
#define DW(c) (((c) & 1) * 4 - 2)
#define PLAN 2
#define SKIP(l, i) 0
#define MINL0 2
#define MINL1 3
#define PUBD 1
#define ELAG (PUBD + 2)
#define NL "\n\t"
constexpr int NR = 8;
constexpr int MS = 1024, KD = 8192, ND = 8192, NS_OUT = 1024;
constexpr int BM = 256, BN = 128, BK = 64;
constexpr int A_BYTES = BM * BK * 2, B_BYTES = BK * BN * 2, STAGE_BYTES = A_BYTES + B_BYTES;
constexpr int RING_BYTES = 3 * STAGE_BYTES;
constexpr int LDS_BYTES = RING_BYTES + 2 * 8192;
constexpr unsigned long long DEADLINE = 50000000ull;
constexpr int AUX_SYS = 17;
static_assert((BETA + DELTA) % 2 == 0 && BETA + DELTA <= 24, "generation 0 must keep an even, positive length");

struct Params { const void* x; const void* w; void* out[NR]; unsigned* ws[NR]; int rank; unsigned tab[256]; };
constexpr int WS_TMO = 0;
constexpr int WS_CNT = 32;
constexpr int WS_DUMP = WS_CNT + 256 * 32;
constexpr int WS_DUMP3 = WS_DUMP + 2 * 256 * 32;
constexpr int WS_RECV = WS_DUMP3 + 1024 * 32;
constexpr int WS_END = WS_RECV + 2560 * 32;
constexpr size_t PART_OFF = 1u << 20;
constexpr unsigned SLAB = 65536u;
constexpr unsigned PART_BYTES = 3u * 256u * SLAB;
static_assert(WS_END <= 163840, "flags area");

constexpr float GK1 = -2.0f * 0.7978845608028654f * 1.4426950408889634f, GK2 = GK1 * 0.044715f;
__device__ __forceinline__ unsigned gelu_pk(f32x2 y) {
  const f32x2 u = __builtin_elementwise_fma(y * y, (f32x2){GK2, GK2}, (f32x2){GK1, GK1}) * y;
  const f32x2 d = (f32x2){__builtin_amdgcn_exp2f(u.x), __builtin_amdgcn_exp2f(u.y)} + 1.0f;
  const f32x2 r = y * (f32x2){__builtin_amdgcn_rcpf(d.x), __builtin_amdgcn_rcpf(d.y)};
  return __builtin_bit_cast(unsigned, __builtin_convertvector(r, bf16x2));
}
__device__ __forceinline__ float e_add(float a, float b) { float r; asm volatile("v_add_f32 %0, %1, %2" NL : "=v"(r) : "v"(a), "v"(b)); return r; }
__device__ __forceinline__ float e_lo(unsigned a) { float r; asm volatile("v_lshlrev_b32 %0, 16, %1" NL : "=v"(r) : "v"(a)); return r; }
__device__ __forceinline__ float e_hi(unsigned a) { float r; asm volatile("v_and_b32 %0, 0xffff0000, %1" NL : "=v"(r) : "v"(a)); return r; }
__device__ __forceinline__ unsigned e_gelu2(float y0, float y1, float k1v) {
  const float k2 = -2.0f * 0.7978845608028654f * 1.4426950408889634f * 0.044715f;
  float t0, t1; unsigned r;
  asm volatile("v_mul_f32 %1, %3, %3" NL "v_mul_f32 %2, %4, %4" NL "v_fma_f32 %1, %1, %5, %6" NL "v_fma_f32 %2, %2, %5, %6" NL "v_mul_f32 %1, %1, %3" NL "v_mul_f32 %2, %2, %4" NL
               "v_exp_f32 %1, %1" NL "v_exp_f32 %2, %2" NL "v_add_f32 %1, 1.0, %1" NL "v_add_f32 %2, 1.0, %2" NL "v_rcp_f32 %1, %1" NL "v_rcp_f32 %2, %2" NL
               "v_mul_f32 %1, %1, %3" NL "v_mul_f32 %2, %2, %4" NL "v_cvt_pk_bf16_f32 %0, %1, %2" NL : "=v"(r), "=&v"(t0), "=&v"(t1) : "v"(y0), "v"(y1), "s"(k2), "v"(k1v));
  return r;
}
#define BAR() asm volatile("s_barrier" ::: "memory")
#define WAITV(n) asm volatile("s_waitcnt vmcnt(" #n ")" ::: "memory")
#define SCHED0() __builtin_amdgcn_sched_barrier(0)
#define RFL(x) __builtin_amdgcn_readfirstlane(x)
#define PIN(x) asm volatile("" : "+v"(x))

#define STAMPAT(i)
#define STAMPOUT(base)

__device__ __forceinline__ bool wait_flag(const unsigned* flag, unsigned call, unsigned* tmo) {
  const unsigned long long t0 = __builtin_amdgcn_s_memrealtime();
  bool bad = false;
  unsigned spins = 0;
  while (__hip_atomic_load(flag, __ATOMIC_RELAXED, __HIP_MEMORY_SCOPE_SYSTEM) < call) {
    if ((++spins & 15) == 0 && (__hip_atomic_load(tmo, __ATOMIC_RELAXED, __HIP_MEMORY_SCOPE_AGENT) == call || __builtin_amdgcn_s_memrealtime() - t0 > DEADLINE)) {
      __hip_atomic_store(tmo, call, __ATOMIC_RELAXED, __HIP_MEMORY_SCOPE_AGENT);
      bad = true; break;
    }
    __builtin_amdgcn_s_sleep(1);
  }
  return bad;
}

__global__ __launch_bounds__(512, 2) void gemm_a2a(Params p) {
  __shared__ __attribute__((aligned(1024))) char smem[LDS_BYTES];
  LAS char* lds = (LAS char*)smem;
  const unsigned ldsb = (unsigned)(size_t)lds;
  const int tid = threadIdx.x;
  const int wid = RFL(tid >> 6);
  const int lane = tid & 63;
  const int b = blockIdx.x;
  const int xj = b & 7, idx = b >> 3, q = xj & 3;
  const int vb = b ^ ((xj & 1) << 7);
  const bool ltype = vb >= 128;
  const unsigned tab = p.tab[b];
  unsigned* const myws = p.ws[p.rank];
  const int cx = 6 ^ (2 * q);
  const int dsh = DW(idx >> 2);
  const int G0 = 64 + ((xj & 1) ? -dsh : dsh);
  const int G1 = 96 + ((xj & 1) ? -BETA : BETA) + (ltype ? DELTA : -DELTA);
  const int n2 = 32, s2 = q * 32;
  const int NST = G1 + n2;

  if (wid >= 6) {
    const int e = wid - 6;
    unsigned call = RFL(__hip_atomic_load(myws + WS_CNT + b * 32, __ATOMIC_RELAXED, __HIP_MEMORY_SCOPE_AGENT)) + 1;
    int nb = NST + 2;
#define STEP() do { if (nb > 0) { BAR(); --nb; } SCHED0(); } while (0)
    auto rp = __builtin_amdgcn_make_buffer_rsrc((char*)myws + PART_OFF, 0, PART_BYTES, 0x00020000);
    const unsigned vld = (unsigned)((lane >> 5) * 1024 + ((lane >> 3) & 3) * 256 + (lane & 7) * 16);
    const unsigned vrd = ldsb + RING_BYTES + e * 8192 + (unsigned)(((lane >> 3) & 1) * 4096 + (((lane >> 2) & 1) * 4 + (lane & 1) * 2) * 128 + (lane >> 4) * 16 + ((lane >> 1) & 1) * 8);
    const unsigned vst = (unsigned)((lane >> 4) * (NS_OUT * 2) + (lane & 15) * 16);
    bool bad = false;
    float k1v = -2.0f * 0.7978845608028654f * 1.4426950408889634f; PIN(k1v);
    const unsigned* fl1 = myws + WS_DUMP + (256 + b - q + (lane & 3)) * 32;
    auto wait_gen = [&](const unsigned* fl) {
      const unsigned long long t0 = __builtin_amdgcn_s_memrealtime();
      for (;;) {
        unsigned v, tv;
        asm volatile("global_load_dword %0, %2, off sc0 sc1\n\tglobal_load_dword %1, %3, off sc1" : "=&v"(v), "=&v"(tv) : "v"(fl), "v"(myws + WS_TMO) : "memory");
        STEP();
        asm volatile("s_waitcnt vmcnt(0)" : "+v"(v), "+v"(tv) :: "memory");
        if (__all(v >= call)) break;
        if (RFL(tv) == call || __builtin_amdgcn_s_memrealtime() - t0 > DEADLINE) {
          if (lane == 0) __hip_atomic_store(myws + WS_TMO, call, __ATOMIC_RELAXED, __HIP_MEMORY_SCOPE_AGENT);
          bad = true; break;
        }
        if (nb == 0) __builtin_amdgcn_s_sleep(1);
      }
    };
    auto row8 = [&](int U) { return U < 8 ? (xj & 1) * 16 + U * 2 + e : q * 8 + (U - 8) * 2 + e; };
    auto fetch = [&](int U) {
      const int r8 = row8(U);
      const unsigned sld = (unsigned)((U < 8 ? b & ~1 : 256 + b - q) * SLAB + (r8 >> 4) * 32768 + ((r8 >> 1) & 7) * 2048 + (r8 & 1) * 128);
#pragma unroll
      for (int s = 0; s < 4; ++s)
        __builtin_amdgcn_raw_ptr_buffer_load_lds(rp, (LAS void*)(lds + RING_BYTES + e * 8192 + (s >> 1) * 4096 + (s & 1) * 1024), 16, vld, sld + (s & 1) * SLAB + (s >> 1) * 16384, 0, AUX_SYS);
      if (U >= 8) {
#pragma unroll
        for (int s = 0; s < 4; ++s)
          __builtin_amdgcn_raw_ptr_buffer_load_lds(rp, (LAS void*)(lds + RING_BYTES + e * 8192 + (s >> 1) * 4096 + 2048 + (s & 1) * 1024), 16, vld, sld + (2 + (s & 1)) * SLAB + (s >> 1) * 16384, 0, AUX_SYS);
      }
    };
    const int skip = SKIP(ltype, idx);
    bool have = false;
    while (nb > NST - (G0 < 64 ? 128 - G0 : G0) - ELAG) STEP();
    for (int U = 0; U < 12; ++U) {
      int len = 4;
      if (!have) {
        wait_gen(U == 0 ? myws + WS_DUMP + ((b & ~1) + (lane & 1)) * 32 : fl1);
        if (U == 0) STAMPAT(0); else STAMPAT(2);
        fetch(U);
        STEP();
        len = 4 - ((NST + 2 - nb + ((idx >> 2) & 3)) & 3);
        if (len < (U == 0 ? MINL0 : MINL1)) { for (int i = 0; i < len; ++i) STEP(); len = 4; }
        WAITV(0);
      } else {
        if ((skip >> U) & 1) { STEP(); STEP(); STEP(); STEP(); }
        WAITV(2);
      }
      u32x2 L[2][2][2]; u32x4 o[2];
      float y[2][8];
#define RD8(off) asm volatile("ds_read_b64 %0, %8\n\tds_read_b64 %1, %8 offset:128\n\tds_read_b64 %2, %8 offset:1024\n\tds_read_b64 %3, %8 offset:1152\n\t" \
                   "ds_read_b64 %4, %8 offset:64\n\tds_read_b64 %5, %8 offset:192\n\tds_read_b64 %6, %8 offset:1088\n\tds_read_b64 %7, %8 offset:1216\n\ts_waitcnt lgkmcnt(0)" \
                   : "=&v"(L[0][0][0]), "=&v"(L[0][0][1]), "=&v"(L[0][1][0]), "=&v"(L[0][1][1]), "=&v"(L[1][0][0]), "=&v"(L[1][0][1]), "=&v"(L[1][1][0]), "=&v"(L[1][1][1]) : "v"(vrd + (off)) : "memory")
      unsigned vf = 0;
      if (U == 7) asm volatile("global_load_dword %0, %1, off sc0 sc1" : "=v"(vf) : "v"(fl1) : "memory");
      RD8(0);
#pragma unroll
      for (int h = 0; h < 2; ++h)
#pragma unroll
        for (int d = 0; d < 4; ++d) {
          const float a0 = e_lo(L[h][0][d >> 1][d & 1]), a1 = e_hi(L[h][0][d >> 1][d & 1]), c0 = e_lo(L[h][1][d >> 1][d & 1]), c1 = e_hi(L[h][1][d >> 1][d & 1]);
          y[h][2 * d] = e_add(a0, c0); y[h][2 * d + 1] = e_add(a1, c1);
        }
      if (U >= 8) RD8(2048);
      if (len > 3) STEP();
      have = U < 11;
      if (U == 7) { asm volatile("s_waitcnt vmcnt(0)" : "+v"(vf) :: "memory"); have = __all(vf >= call); }
      if (U >= 8) {
#pragma unroll
        for (int h = 0; h < 2; ++h)
#pragma unroll
          for (int d = 0; d < 4; ++d)
#pragma unroll
            for (int s = 0; s < 2; ++s) {
              const float a0 = e_lo(L[h][s][d >> 1][d & 1]), a1 = e_hi(L[h][s][d >> 1][d & 1]);
              y[h][2 * d] = e_add(y[h][2 * d], a0); y[h][2 * d + 1] = e_add(y[h][2 * d + 1], a1);
            }
      }
      SCHED0();
      if (have && e == 0) fetch(U + 1);
      if (len > 2) STEP();
#pragma unroll
      for (int d = 0; d < 4; ++d) o[0][d] = e_gelu2(y[0][2 * d], y[0][2 * d + 1], k1v);
      SCHED0();
      if (have && e == 1) fetch(U + 1);
      STEP();
#pragma unroll
      for (int d = 0; d < 4; ++d) o[1][d] = e_gelu2(y[1][2 * d], y[1][2 * d + 1], k1v);
      if (bad) o[0] = o[1] = (u32x4){0x7fc07fc0u, 0x7fc07fc0u, 0x7fc07fc0u, 0x7fc07fc0u};
      const unsigned te = (tab >> (U < 8 ? 0 : 8)) & 0xff;
      auto ro = __builtin_amdgcn_make_buffer_rsrc(p.out[(te >> 3) & 7], 0, 8192 * NS_OUT * 2, 0x00020000);
      const unsigned so = (unsigned)((p.rank * MS + (te >> 6) * BM + row8(U) * 8) * (NS_OUT * 2) + (te & 7) * (BN * 2));
      __builtin_amdgcn_raw_buffer_store_b128(o[0], ro, vst, so, AUX_SYS);
      __builtin_amdgcn_raw_buffer_store_b128(o[1], ro, vst, so + 4 * (NS_OUT * 2), AUX_SYS);
      asm volatile("s_nop 1");
      STEP();
      if (U == 7) STAMPAT(1);
    }
    STAMPAT(3);
    STEP();
    WAITV(0);
    STAMPAT(4);
    const unsigned dl = (tab >> (8 * (lane & 3) + 3)) & 7;
    const int slot = (b * 10 + lane * 2 + e) * 32;
    if (lane < 2 && dl != (unsigned)p.rank) __hip_atomic_store(p.ws[dl] + WS_RECV + slot, call, __ATOMIC_RELAXED, __HIP_MEMORY_SCOPE_SYSTEM);
    while (nb > 0) STEP();
    STAMPAT(5);
    if (lane < 2) {
      if (dl != (unsigned)p.rank) {
        if (wait_flag(myws + WS_RECV + slot, call, myws + WS_TMO))
          __hip_atomic_store((unsigned short*)p.out[p.rank] + (size_t)(p.rank * MS) * NS_OUT, (unsigned short)0x7fc0, __ATOMIC_RELAXED, __HIP_MEMORY_SCOPE_SYSTEM);
      }
    }
    if (e == 0 && lane == 0) __hip_atomic_store(myws + WS_CNT + b * 32, call, __ATOMIC_RELAXED, __HIP_MEMORY_SCOPE_AGENT);
    STAMPAT(6);
    if (e == 0) STAMPOUT(180224);
  } else if (wid >= 4) {
    const int dw = wid - 4;
    unsigned vcall;
    asm volatile("global_load_dword %0, %1, off sc1" : "=v"(vcall) : "v"(myws + WS_CNT + b * 32) : "memory");
    constexpr int NA = 16, NB = 8;
    auto rx = __builtin_amdgcn_make_buffer_rsrc((char*)p.x - 4096, 0, MS * KD * 2 + 4096, 0x00020000);
    auto rw = __builtin_amdgcn_make_buffer_rsrc((void*)p.w, 0, KD * ND * 2, 0x00020000);
    unsigned vA[NA], vB[NB];
#pragma unroll
    for (int j = 0; j < NB; ++j) { vB[j] = (unsigned)((4 * (dw * NB + j) + (lane >> 4)) * (ND * 2) + (((lane & 15) ^ (((lane >> 4) | (((j >> 1) & 1) << 2)) << 1)) * 16) - (j & 3) * 1024); PIN(vB[j]); }
    auto calcB = [&](int t) -> unsigned {
      t = t < NST ? t : NST - 1;
      const int g = t < G0 ? 0 : t < G1 ? 1 : 2;
      const unsigned te = tab >> (8 * g);
      const int ph = g == 0 ? (int)(tab >> 24) & 3 : (idx >> 2) & 3, lo = g == 0 ? 0 : g == 1 ? G0 : G1, hi = g == 0 ? G0 : g == 1 ? G1 : NST, bs = ((t + ph) & ~3) - ph;
      if (bs >= lo && bs + 4 <= hi) t = ((t + ph) ^ ((te >> 6) & 3)) - ph;
      const int u = t - lo, v = g == 0 ? u : 32 + (hi - lo) - 1 - u;
      const int k = ((xj >> 2) * 32 + (g < 2 ? (xj & 2) * 32 + ((xj & 1) ? -1 - v : v) : 32 + s2 + ((xj & 1) ? n2 - 1 - u : u))) & 127;
      return (te & 63) * (BN * 2) + (unsigned)k * (BK * ND * 2) + ((te & 0xc0) << 22);
    };
    unsigned vt0 = calcB(lane), vt1 = 0, vt2 = 0;
    auto offB = [&](int t) -> unsigned {
      const unsigned u0 = __builtin_amdgcn_readlane(vt0, t & 63), u1 = __builtin_amdgcn_readlane(vt1, t & 63), u2 = __builtin_amdgcn_readlane(vt2, t & 63);
      return t < 64 ? u0 : t < 128 ? u1 : u2;
    };
    auto b2a = [&](unsigned sB) -> unsigned { return (sB >> 28) * (BM * KD * 2) + ((sB >> 20) & 127) * (BK * 2); };
#define Q4(r, l, v, i, s) do { __builtin_amdgcn_raw_ptr_buffer_load_lds(r, (LAS void*)(l), 16, v[i], s, 0, 0); __builtin_amdgcn_raw_ptr_buffer_load_lds(r, (LAS void*)(l), 16, v[i + 1], s, 1024, 0); \
  __builtin_amdgcn_raw_ptr_buffer_load_lds(r, (LAS void*)(l), 16, v[i + 2], s, 2048, 0); __builtin_amdgcn_raw_ptr_buffer_load_lds(r, (LAS void*)(l), 16, v[i + 3], s, 3072, 0); } while (0)
    auto issueB = [&](unsigned sB, unsigned lb) {
      sB &= 0x0fffffffu;
#pragma unroll
      for (int j = 0; j < NB; j += 4) Q4(rw, (size_t)(lb + A_BYTES + (dw * NB + j) * 1024), vB, j, sB);
    };
    auto issueA = [&](unsigned sA, unsigned lb) {
#pragma unroll
      for (int i = 0; i < NA; i += 4) Q4(rx, (size_t)(lb + (dw * NA + i) * 1024), vA, i, sA);
    };
    issueB(offB(0), ldsb + 2 * STAGE_BYTES);
    SCHED0();
#pragma unroll
    for (int i = 0; i < NA; ++i) { vA[i] = (unsigned)((8 * ((dw * NA + i) ^ (2 * cx)) + (lane >> 3)) * (KD * 2) + 4096 + (((lane & 7) ^ ((4 * (i & 1) + (lane >> 4)) & 7)) * 16) - (i & 3) * 1024); PIN(vA[i]); }
    issueA(b2a(offB(0)), ldsb);
    SCHED0();
    vt1 = calcB(64 + lane); vt2 = calcB(128 + lane); PIN(vt1); PIN(vt2);
    issueB(offB(1), ldsb); issueA(b2a(offB(1)), ldsb + STAGE_BYTES); issueB(offB(2), ldsb + STAGE_BYTES);
    asm volatile("s_waitcnt vmcnt(24)" : "+v"(vcall) :: "memory");
    BAR();
    vcall += 1;
    BAR();
    issueA(b2a(offB(2)), ldsb + 2 * STAGE_BYTES); issueB(offB(3), ldsb + 2 * STAGE_BYTES);
    unsigned lb = ldsb;
    unsigned sB = offB(4), sA = b2a(offB(3));
    int pub = G0 + PUBD;
    for (int s = 0; s < NST - 3; ++s) {
      WAITV(24); BAR();
      if (s == pub) {
        if (dw == 0 && lane == 0)
          __hip_atomic_store(myws + WS_DUMP + (s == G0 + PUBD ? b : 256 + b) * 32, vcall, __ATOMIC_RELAXED, __HIP_MEMORY_SCOPE_SYSTEM);
        pub = G1 + PUBD;
      }
      issueA(sA, lb);
      SCHED0();
      unsigned nA = b2a(sB), nB = offB(s + 5);
      asm volatile("" : "+s"(nB), "+s"(nA));
      SCHED0();
      issueB(sB, lb);
      sA = nA; sB = nB;
      lb = lb == ldsb + 2 * STAGE_BYTES ? ldsb : lb + STAGE_BYTES;
    }
    WAITV(24); BAR();
    WAITV(0); BAR();
    BAR();
  } else {
    unsigned vcall;
    asm volatile("global_load_dword %0, %1, off sc1" : "=v"(vcall) : "v"(myws + WS_CNT + b * 32) : "memory");
    const int wm = wid >> 1, wn = wid & 1;
    unsigned aoff[2], boff[4];
    {
      const int l15 = lane & 15, g = lane >> 4;
#pragma unroll
      for (int ks = 0; ks < 2; ++ks) aoff[ks] = ldsb + (unsigned)((wm * 128 + l15) * 128 + (((ks * 4 + g) ^ (l15 >> 1)) * 16));
      const int qq = l15 >> 2, pp = lane & 3, h = qq | ((g & 1) << 2);
#pragma unroll
      for (int ni = 0; ni < 4; ++ni) boff[ni] = ldsb + (unsigned)(2 * STAGE_BYTES + A_BYTES + (8 * g + qq) * 256 + (((wn * 4 + ni) ^ h) * 32) + 8 * pp);
    }
    f32x4 acc[8][4];
    bf16x8 a[4][2]; bf16x4 bb[2][4][2][2];
    int bufn = 0; unsigned dlt;

#define RDA(slot, ks, mi) a[slot][ks] = *(LAS bf16x8*)(size_t)(aoff[ks] + (mi) * 2048)
#define RDB(P, ni, ks, t) bb[P][ni][ks][t] = __builtin_amdgcn_ds_read_tr16_b64_v4bf16((LAS bf16x4*)(size_t)(boff[ni] + (ks) * 8192 + (t) * 1024))
#define BF(P, ni, ks) __builtin_shufflevector(bb[P][ni][ks][0], bb[P][ni][ks][1], 0, 1, 2, 3, 4, 5, 6, 7)
#define MF(ZERO, P, mi, ni, ks) do { if (ZERO) asm volatile("v_mfma_f32_16x16x32_bf16 %0, %1, %2, 0" : "+v"(acc[mi][ni]) : "v"(BF(P, ni, ks)), "v"(a[(mi) & 3][ks])); \
   else asm volatile("v_mfma_f32_16x16x32_bf16 %0, %1, %2, %0" : "+v"(acc[mi][ni]) : "v"(BF(P, ni, ks)), "v"(a[(mi) & 3][ks])); } while (0)
#define CVT(k, i, T) asm volatile("v_cvt_pk_bf16_f32 %0, %1, %2" : "=v"(T[i]) : "v"(acc[(k) >> 1][2 * ((k) & 1) + ((i) >> 1)][2 * ((i) & 1)]), "v"(acc[(k) >> 1][2 * ((k) & 1) + ((i) >> 1)][2 * ((i) & 1) + 1]))
#define PUT(k, T) __builtin_amdgcn_raw_buffer_store_b128((u32x4){T[0], T[1], T[2], T[3]}, rd, vl + ((k) & 1) * 1024, sd + (unsigned)((((k) >> 1) ^ cx) * 2048), AUX_SYS)
#define STAGE(KIND, P, DRAIN) do { \
    _Pragma("unroll") for (int mi = 0; mi < 8; ++mi) { \
      _Pragma("unroll") for (int j = 0; j < 8; ++j) { \
        MF(KIND == 1 && (j & 1) == 0, P, mi, j >> 1, j & 1); SCHED0(); \
        if (j == 0) RDA((mi + 3) & 3, 0, (mi + 3) & 7); \
        if (j == (mi == 4 ? 2 : 4)) RDA((mi + 3) & 3, 1, (mi + 3) & 7); \
        if (mi < 4 && (j == 1 || j == 2 || j == 5 || j == 6)) { const int r_ = (j < 4 ? j - 1 : j - 3); RDB((P) ^ 1, mi, r_ >> 1, r_ & 1); } \
        if (mi == 3 && j == 3) { bufn = bufn == 2 ? 0 : bufn + 1; dlt = bufn == 0 ? (unsigned)(-2 * STAGE_BYTES) : (unsigned)STAGE_BYTES; asm volatile("" : "+s"(dlt), "+s"(bufn)); } \
        if (mi == 4 && j == 4) { aoff[0] += dlt; PIN(aoff[0]); } \
        if (mi == 4 && j == 5) { aoff[1] += dlt; PIN(aoff[1]); } \
        if (mi == 4 && j == 7) { if (DRAIN) WAITV(0); BAR(); } \
        if (mi == 5 && j == 2) { boff[0] += dlt; PIN(boff[0]); } \
        if (mi == 5 && j == 6) { boff[1] += dlt; PIN(boff[1]); } \
        if (mi == 6 && j == 2) { boff[2] += dlt; PIN(boff[2]); } \
        if (mi == 6 && j == 6) { boff[3] += dlt; PIN(boff[3]); } \
        if (KIND == 2 && mi >= 1) { if (j < 4) CVT(mi - 1, j & 3, t0); if (j == 5) PUT(mi - 1, t0); } \
        if (KIND == 1) { \
          if (mi < 7) { if (j < 4) CVT(mi + 7, j & 3, t0); if (j == 5) PUT(mi + 7, t0); } \
          if (mi == 6 && j >= 4) CVT(14, j & 3, t1); \
          if (mi == 7) { if (j < 4) CVT(15, j & 3, t0); if (j == 1) PUT(14, t1); if (j == 6) PUT(15, t0); } \
        } \
        SCHED0(); \
      } \
    } } while (0)

    __builtin_amdgcn_s_setprio(2);
    STAMPAT(0);
    BAR();
    STAMPAT(1);
#pragma unroll
    for (int r = 0; r < 16; ++r) RDB(0, r >> 2, (r >> 1) & 1, r & 1);
#pragma unroll
    for (int mi = 0; mi < 3; ++mi) { RDA(mi, 0, mi); RDA(mi, 1, mi); }
#pragma unroll
    for (int ni = 0; ni < 4; ++ni) { boff[ni] -= 2 * STAGE_BYTES; PIN(boff[ni]); }
    asm volatile("s_waitcnt vmcnt(0)" : "+v"(vcall) :: "memory");
    const unsigned call = RFL(vcall) + 1;
    SCHED0();
    BAR();
    SCHED0();
    auto rp = __builtin_amdgcn_make_buffer_rsrc((char*)myws + PART_OFF, 0, PART_BYTES, 0x00020000);
    auto rd = __builtin_amdgcn_make_buffer_rsrc((char*)myws + PART_OFF, 0, 0, 0x00020000);
    unsigned vl = lane * 16; PIN(vl);
    unsigned sd = 0, t0[4], t1[4];
#pragma unroll
    for (int mi = 0; mi < 8; ++mi)
#pragma unroll
      for (int ni = 0; ni < 4; ++ni) asm volatile("" : "=v"(acc[mi][ni]));
    for (int g = 0; g < 3; ++g) {
      STAGE(1, 0, false);
      const int n = g == 0 ? G0 : g == 1 ? G1 - G0 : n2;
      for (int s = 2; s < n; s += 2) {
        STAGE(0, 1, PUBD == 1);
        STAGE(0, 0, PUBD == 2);
      }
      rd = rp;
      sd = (unsigned)((g * 256 + (g < 2 ? b : vb)) * SLAB + wid * 16384);
      STAGE(2, 1, false);
    }
    asm volatile("s_nop 7");
    __builtin_amdgcn_s_setprio(0);
    STAMPAT(2);

    const int ln = __builtin_amdgcn_mbcnt_hi(-1, __builtin_amdgcn_mbcnt_lo(-1, 0));
    const int v0 = vb - q;
#pragma unroll
    for (int k = 7; k < 12; ++k) {
      unsigned t[4];
#pragma unroll
      for (int i = 0; i < 4; ++i) CVT(k, i, t);
      PUT(k, t); asm volatile("s_nop 1" :: "v"(t[0]), "v"(t[1]), "v"(t[2]), "v"(t[3]));
    }
    WAITV(0);
    STAMPAT(3);
    bool bad = false;
    if (ln == 0) __hip_atomic_store(myws + WS_DUMP3 + (vb * 4 + wid) * 32, call, __ATOMIC_RELAXED, __HIP_MEMORY_SCOPE_SYSTEM);
    if (ln < 4 && ln != q) bad = wait_flag(myws + WS_DUMP3 + ((v0 + ln) * 4 + wid) * 32, call, myws + WS_TMO);
    bad = __any(bad);
    asm volatile("" ::: "memory");
    STAMPAT(4);
    f32x4 own[2][4];
    {
      u32x4 L[3][2][2];
#pragma unroll
      for (int s = 0; s < 3; ++s) {
        const int pq = (q + 1 + s) & 3;
        const unsigned sl = (unsigned)((2 * 256 + v0 + pq) * SLAB + wid * 16384 + q * 4096);
#pragma unroll
        for (int m2 = 0; m2 < 2; ++m2)
#pragma unroll
          for (int h = 0; h < 2; ++h) L[s][m2][h] = __builtin_amdgcn_raw_buffer_load_b128(rp, ln * 16 + h * 1024, sl + m2 * 2048, AUX_SYS);
      }
#pragma unroll
      for (int m2 = 0; m2 < 2; ++m2)
#pragma unroll
        for (int ni = 0; ni < 4; ++ni) {
          own[m2][ni] = acc[6 + m2][ni];
#pragma unroll
          for (int s = 0; s < 3; ++s) {
            const unsigned u0 = L[s][m2][ni >> 1][(ni & 1) * 2], u1 = L[s][m2][ni >> 1][(ni & 1) * 2 + 1];
            own[m2][ni] += (f32x4){__builtin_bit_cast(float, u0 << 16), __builtin_bit_cast(float, u0 & 0xffff0000u), __builtin_bit_cast(float, u1 << 16), __builtin_bit_cast(float, u1 & 0xffff0000u)};
          }
        }
    }
    const unsigned te = (tab >> 16) & 0xff;
    const int dest = (te >> 3) & 7;
    {
      LAS char* st = lds + wid * 4096;
      const int l15 = ln & 15, g = ln >> 4;
      const unsigned wo = (unsigned)(l15 * 128 + (g & 1) * 8);
      auto ro = __builtin_amdgcn_make_buffer_rsrc(p.out[dest], 0, 8192 * NS_OUT * 2, 0x00020000);
      const unsigned vo = (unsigned)((ln >> 3) * (NS_OUT * 2) + (((ln & 7) ^ (ln >> 3)) * 16));
      const unsigned so = (unsigned)((p.rank * MS + (te >> 6) * BM + wm * 128 + q * 32) * (NS_OUT * 2) + (te & 7) * (BN * 2) + wn * 128);
      u32x4 r[4];
#pragma unroll
      for (int m2 = 0; m2 < 2; ++m2) {
#pragma unroll
        for (int ni = 0; ni < 4; ++ni) {
          f32x4 v = own[m2][ni];
          u32x2 o; o.x = gelu_pk((f32x2){v[0], v[1]}); o.y = gelu_pk((f32x2){v[2], v[3]});
          if (bad) o = (u32x2){0x7fc07fc0u, 0x7fc07fc0u};
          *(LAS u32x2*)(st + m2 * 2048 + wo + (((ni * 2 + (g >> 1)) ^ (l15 & 7)) * 16)) = o;
        }
        __builtin_amdgcn_fence(__ATOMIC_RELEASE, "wavefront"); __builtin_amdgcn_wave_barrier(); __builtin_amdgcn_fence(__ATOMIC_ACQUIRE, "wavefront");
        r[2 * m2] = *(LAS u32x4*)(st + m2 * 2048 + ln * 16); r[2 * m2 + 1] = *(LAS u32x4*)(st + m2 * 2048 + 1024 + ln * 16);
        if (m2 == 0) { __builtin_amdgcn_raw_buffer_store_b128(r[0], ro, vo, so, AUX_SYS); SCHED0(); asm volatile("s_nop 1"); SCHED0(); }
      }
#pragma unroll
      for (int j = 1; j < 4; ++j) {
        if (dest != p.rank) WAITV(0);
        __builtin_amdgcn_raw_buffer_store_b128(r[j], ro, vo, so + j * 8 * (NS_OUT * 2), AUX_SYS);
      }
    }
    STAMPAT(5);
    WAITV(0);
    STAMPAT(6);
    if (ln == 0 && dest != p.rank) {
      const int slot = (b * 10 + 6 + wid) * 32;
      __hip_atomic_store(p.ws[dest] + WS_RECV + slot, call, __ATOMIC_RELAXED, __HIP_MEMORY_SCOPE_SYSTEM);
      if (wait_flag(myws + WS_RECV + slot, call, myws + WS_TMO))
        __hip_atomic_store((unsigned short*)p.out[p.rank] + (size_t)(p.rank * MS) * NS_OUT, (unsigned short)0x7fc0, __ATOMIC_RELAXED, __HIP_MEMORY_SCOPE_SYSTEM);
    }
    STAMPAT(7);
    if (wid == 0) STAMPOUT(163840);
  }
}

extern "C" void kernel_launch(int rank, int nranks, ncclComm_t comm, hipStream_t stream,
                              void* const* d_in, const int* in_sizes, int n_in,
                              void* d_out, int out_size, const int* in_dtypes, int out_dtype,
                              void* const* d_out_all, void* const* d_ws_all, size_t ws_bytes) {
  if (nranks != NR || (unsigned)rank >= (unsigned)NR || ws_bytes < PART_OFF + (size_t)PART_BYTES) return;
  Params p;
  p.x = d_in[0]; p.w = d_in[1]; p.rank = rank;
  for (int i = 0; i < NR; ++i) { p.out[i] = d_out_all[i]; p.ws[i] = (unsigned*)d_ws_all[i]; }
  static const int label[8] = {0, 4, 1, 2, 5, 6, 3, 7};
  int peer[8]; for (int r = 0; r < 8; ++r) peer[label[rank] ^ label[r]] = r;
  unsigned char lst[8][32], phs[8][32]; int cnt[8] = {0}, next[8] = {0}, nextb[8] = {0}, nq = 0;
  auto add = [&](int grp, int delta, int tn, int tm, int ph) { phs[grp][cnt[grp]] = (unsigned char)ph; lst[grp][cnt[grp]++] = (unsigned char)(peer[delta] * 8 + tn + tm * 64); };
#define X 16
  static const unsigned char dl0[64] = {9,9,9,1,2,3,1,2, 3,1,2,3,1,2,3,1, 4,5,6,7,4,5,6,7, 4,5,6,7,4,5,6,7,
                                        1+X,2+X,3+X,2,1+X,2+X,3+X,3, 4+X,5+X,6+X,7+X,4+X,5+X,6+X,7+X,
                                        1,2,3,9,0,0,0,0+X, 4,5,6,7,0,0,0,0+X};
  static const unsigned char dl1[64] = {9,9,1,2,3,1,2,3, 1,2,3,1,2,3,1,2, 4,5,6,7,4,5,6,7, 4,5,6,7,4,5,6,7,
                                        1+X,2+X,3+X,9,1+X,2+X,3+X,3, 4+X,5+X,6+X,7+X,4+X,5+X,6+X,7+X,
                                        1,2,3,9,0,0,0,0+X, 4,5,6,7,0,0,0,0+X};
  static const unsigned char dl2[64] = {9,9,1+X,2,3+X,1,2+X,3, 1+X,2+X,3+X,1,2,3,1,2, 4+X,5+X,6+X,7,4,5,6,7+X, 4,5,6,7+X,4+X,5+X,6+X,7,
                                        1,2,3,9,1,2,3,3, 4,5,6,7,4,5,6,7,
                                        1,2,3,9,0,0,0,0+X, 4,5,6,7,0,0,0+X,0};
  static const unsigned char dl3[64] = {9,9,1,2,3,1,2,3, 1,2,3,1,2,3,1,2, 4,5,6,7,4,5,6,7+X, 4,5,6,7+X,4,5,6,7,
                                        1+X,2+X,3+X,9,1+X,2+X,3+X,3, 4+X,5+X,6+X,7,4+X,5+X,6+X,7,
                                        1,2,3,9,0,0,0,0+X, 4,5,6,7,0,0,0,0+X};
#undef X
  const unsigned char* dl = PLAN == 0 ? dl0 : PLAN == 1 ? dl1 : PLAN == 2 ? dl2 : dl3;
  static const unsigned char good[6] = {0, 2, 3, 4, 6, 7};
  for (int i = 0; i < 64; ++i) {
    const int d0 = dl[i] & 15;
    if (d0 == 9) {
      for (int tm = 0; tm < 4; ++tm) add(i >> 3, 4 + ((tm + nq) & 3), 7, tm, (tm + nq) & 3);
      ++nq;
    } else {
      const int tn = (dl[i] & 16) ? 1 + 4 * nextb[d0]++ : good[next[d0]++];
      for (int tm = 0; tm < 4; ++tm) add(i >> 3, d0, tn, tm, i & 3);
    }
  }
  for (int b = 0; b < 256; ++b) {
    const int xj = b & 7, i = b >> 3, vi = i ^ ((xj & 1) << 4), h = xj >> 2;
    p.tab[b] = lst[xj >> 1][i] | (lst[4 + h][i] << 8) | (lst[6 + h][vi] << 16) | ((unsigned)phs[xj >> 1][i] << 24);
  }
  gemm_a2a<<<256, 512, 0, stream>>>(p);
}
```

```cpp
#include <hip/hip_runtime.h>
#include <rccl/rccl.h>
#include <cstdio>
#include <type_traits>

#define LAS __attribute__((address_space(3)))
typedef __bf16 bf16x4 __attribute__((ext_vector_type(4)));
typedef __bf16 bf16x8 __attribute__((ext_vector_type(8)));
typedef __bf16 bf16x2 __attribute__((ext_vector_type(2)));
typedef float f32x4 __attribute__((ext_vector_type(4)));
typedef float f32x2 __attribute__((ext_vector_type(2)));
typedef unsigned u32x4 __attribute__((ext_vector_type(4)));
typedef unsigned u32x2 __attribute__((ext_vector_type(2)));

#define BETA 1
#define DELTA 3
#define DW(c) (((c) & 1) * 4 - 2)
#define PLAN 2
#define SKIP(l, i) (xj < 4 ? 2 << ((i) % 7) : 0)
#define MINL0 2
#define MINL1 3
#define PUBD 1
#define BARJ 2
#define LWAIT 1
#define WL " lgkmcnt(0)"
#define ADJ 0x688a6a88u
#define THR_N 0
#define THR_S 4
#define THR_C 17
#define FPACE 0
#define FHALF 0
#define TAB 0
#define XS 0
#define CLK 1
#define DYN 1
#define ROTB 1
#define FARB 1
#define LAMB 84
constexpr int TS0 = 16, TS = 48, MAILS = 80, REP = 9;
#define ELAG (PUBD + 2)
#define NL "\n\t"
constexpr int NR = 8;
constexpr int MS = 1024, KD = 8192, ND = 8192, NS_OUT = 1024;
constexpr int BM = 256, BN = 128, BK = 64;
constexpr int A_BYTES = BM * BK * 2, B_BYTES = BK * BN * 2, STAGE_BYTES = A_BYTES + B_BYTES;
constexpr int RING_BYTES = 3 * STAGE_BYTES;
constexpr int LDS_BYTES = RING_BYTES + 2 * 8192;
constexpr unsigned long long DEADLINE = 50000000ull;
constexpr int AUX_SYS = 17;
static_assert((BETA + DELTA) % 2 == 0 && BETA + DELTA <= 24, "generation 0 must keep an even, positive length");

struct Params { const void* x; const void* w; void* out[NR]; unsigned* ws[NR]; int rank; unsigned tab[256]; };
constexpr int WS_TMO = 0;
constexpr int WS_CNT = 32;
constexpr int WS_DUMP = WS_CNT + 256 * 32;
constexpr int WS_DUMP3 = WS_DUMP + 2 * 256 * 32;
constexpr int WS_RECV = WS_DUMP3 + 1024 * 32;
constexpr int WS_PACE = WS_RECV + 2560 * 32;
constexpr int WS_MAIL = WS_PACE + 8 * 32;
constexpr int WS_END = WS_MAIL + 256 * 64;
constexpr size_t PART_OFF = 1u << 20;
constexpr unsigned SLAB = 65536u;
constexpr unsigned PART_BYTES = 3u * 256u * SLAB;
static_assert(WS_END <= 163840, "flags area");

constexpr float GK1 = -2.0f * 0.7978845608028654f * 1.4426950408889634f, GK2 = GK1 * 0.044715f;
__device__ __forceinline__ unsigned gelu_pk(f32x2 y) {
  const f32x2 u = __builtin_elementwise_fma(y * y, (f32x2){GK2, GK2}, (f32x2){GK1, GK1}) * y;
  const f32x2 d = (f32x2){__builtin_amdgcn_exp2f(u.x), __builtin_amdgcn_exp2f(u.y)} + 1.0f;
  const f32x2 r = y * (f32x2){__builtin_amdgcn_rcpf(d.x), __builtin_amdgcn_rcpf(d.y)};
  return __builtin_bit_cast(unsigned, __builtin_convertvector(r, bf16x2));
}
__device__ __forceinline__ float e_add(float a, float b) { float r; asm volatile("v_add_f32 %0, %1, %2" NL : "=v"(r) : "v"(a), "v"(b)); return r; }
__device__ __forceinline__ float e_lo(unsigned a) { float r; asm volatile("v_lshlrev_b32 %0, 16, %1" NL : "=v"(r) : "v"(a)); return r; }
__device__ __forceinline__ float e_hi(unsigned a) { float r; asm volatile("v_and_b32 %0, 0xffff0000, %1" NL : "=v"(r) : "v"(a)); return r; }
__device__ __forceinline__ unsigned e_gelu2(float y0, float y1, float k1v) {
  const float k2 = -2.0f * 0.7978845608028654f * 1.4426950408889634f * 0.044715f;
  float t0, t1; unsigned r;
  asm volatile("v_mul_f32 %1, %3, %3" NL "v_mul_f32 %2, %4, %4" NL "v_fma_f32 %1, %1, %5, %6" NL "v_fma_f32 %2, %2, %5, %6" NL "v_mul_f32 %1, %1, %3" NL "v_mul_f32 %2, %2, %4" NL
               "v_exp_f32 %1, %1" NL "v_exp_f32 %2, %2" NL "v_add_f32 %1, 1.0, %1" NL "v_add_f32 %2, 1.0, %2" NL "v_rcp_f32 %1, %1" NL "v_rcp_f32 %2, %2" NL
               "v_mul_f32 %1, %1, %3" NL "v_mul_f32 %2, %2, %4" NL "v_cvt_pk_bf16_f32 %0, %1, %2" NL : "=v"(r), "=&v"(t0), "=&v"(t1) : "v"(y0), "v"(y1), "s"(k2), "v"(k1v));
  return r;
}
#define BAR() asm volatile("s_barrier" ::: "memory")
#define WAITV(n) asm volatile("s_waitcnt vmcnt(" #n ")" ::: "memory")
#define SCHED0() __builtin_amdgcn_sched_barrier(0)
#define RFL(x) __builtin_amdgcn_readfirstlane(x)
#define PIN(x) asm volatile("" : "+v"(x))

#define STAMPAT(i)
#define STAMPOUT(base)

__device__ __forceinline__ bool wait_flag(const unsigned* flag, unsigned call, unsigned* tmo) {
  const unsigned long long t0 = __builtin_amdgcn_s_memrealtime();
  bool bad = false;
  unsigned spins = 0;
  while (__hip_atomic_load(flag, __ATOMIC_RELAXED, __HIP_MEMORY_SCOPE_SYSTEM) < call) {
    if ((++spins & 15) == 0 && (__hip_atomic_load(tmo, __ATOMIC_RELAXED, __HIP_MEMORY_SCOPE_AGENT) == call || __builtin_amdgcn_s_memrealtime() - t0 > DEADLINE)) {
      __hip_atomic_store(tmo, call, __ATOMIC_RELAXED, __HIP_MEMORY_SCOPE_AGENT);
      bad = true; break;
    }
    __builtin_amdgcn_s_sleep(1);
  }
  return bad;
}

__global__ __launch_bounds__(512, 2) void gemm_a2a(Params p) {
  __shared__ __attribute__((aligned(1024))) char smem[LDS_BYTES];
  LAS char* lds = (LAS char*)smem;
  const unsigned ldsb = (unsigned)(size_t)lds;
  const int tid = threadIdx.x;
  const int wid = RFL(tid >> 6);
  const int lane = tid & 63;
  const int b = blockIdx.x;
  const int xj = b & 7, idx = b >> 3, q = xj & 3;
  const int vb = b ^ ((xj & 1) << 7);
  const bool ltype = vb >= 128;
  const unsigned tab = p.tab[b];
  unsigned* const myws = p.ws[p.rank];
  const int cx = 6 ^ (2 * q);
  const int dsh = DW(idx >> 2);
  const int G0 = 64 + ((xj & 1) ? -dsh : dsh);
  const int G1 = 96 + ((xj & 1) ? -BETA : BETA) + (ltype ? DELTA : -DELTA);
  int NST = G1 + 32;
    auto calcB = [&](int t, int n2, int s2) -> unsigned {
      const int NST = G1 + n2;
      t = t < NST ? t : NST - 1;
      const int g = t < G0 ? 0 : t < G1 ? 1 : 2;
      const unsigned te = tab >> (8 * g);
      const int ph = g == 0 ? (int)(tab >> 24) & 3 : (idx >> 2) & 3, lo = g == 0 ? 0 : g == 1 ? G0 : G1, hi = g == 0 ? G0 : g == 1 ? G1 : NST, bs = ((t + ph) & ~3) - ph;
      if (ROTB) {
        const int b0 = bs > lo ? bs : lo, m = (bs + 4 < hi ? bs + 4 : hi) - b0;
        int r = t - b0 + (int)((te >> 6) & 3);
        r = m == 1 ? 0 : r >= 2 * m ? r - 2 * m : r >= m ? r - m : r;
        t = b0 + r;
      } else if (bs >= lo && bs + 4 <= hi) t = ((t + ph) ^ ((te >> 6) & 3)) - ph;
      const int u = t - lo, v = g == 0 ? u : 32 + (hi - lo) - 1 - u;
      const int k = ((xj >> 2) * 32 + (g < 2 ? (xj & 2) * 32 + ((xj & 1) ? -1 - v : v) : 32 + s2 + ((xj & 1) ? n2 - 1 - u : u))) & 127;
      return (te & 63) * (BN * 2) + (unsigned)k * (BK * ND * 2) + ((te & 0xc0) << 22);
    };

  if (wid >= 6) {
    const int e = wid - 6;
    unsigned call = RFL(__hip_atomic_load(myws + WS_CNT + b * 32, __ATOMIC_RELAXED, __HIP_MEMORY_SCOPE_AGENT)) + 1;
    int nb = NST + 2;
#define STEP() do { if (nb > 0) { BAR(); --nb; } SCHED0(); } while (0)
    auto rp = __builtin_amdgcn_make_buffer_rsrc((char*)myws + PART_OFF, 0, PART_BYTES, 0x00020000);
    const unsigned vld = (unsigned)((lane >> 5) * 1024 + ((lane >> 3) & 3) * 256 + (lane & 7) * 16);
    const unsigned vrd = ldsb + RING_BYTES + e * 8192 + (unsigned)(((lane >> 3) & 1) * 4096 + (((lane >> 2) & 1) * 4 + (lane & 1) * 2) * 128 + (lane >> 4) * 16 + ((lane >> 1) & 1) * 8);
    const unsigned vst = (unsigned)((lane >> 4) * (NS_OUT * 2) + (lane & 15) * 16);
    bool bad = false;
    float k1v = -2.0f * 0.7978845608028654f * 1.4426950408889634f; PIN(k1v);
    const unsigned* fl1 = myws + WS_DUMP + (256 + b - q + (lane & 3)) * 32;
    auto wait_gen = [&](const unsigned* fl) {
      const unsigned long long t0 = __builtin_amdgcn_s_memrealtime();
      for (;;) {
        unsigned v, tv;
        asm volatile("global_load_dword %0, %2, off sc0 sc1\n\tglobal_load_dword %1, %3, off sc1" : "=&v"(v), "=&v"(tv) : "v"(fl), "v"(myws + WS_TMO) : "memory");
        STEP();
        asm volatile("s_waitcnt vmcnt(0)" : "+v"(v), "+v"(tv) :: "memory");
        if (__all(v >= call)) break;
        if (RFL(tv) == call || __builtin_amdgcn_s_memrealtime() - t0 > DEADLINE) {
          if (lane == 0) __hip_atomic_store(myws + WS_TMO, call, __ATOMIC_RELAXED, __HIP_MEMORY_SCOPE_AGENT);
          bad = true; break;
        }
        if (nb == 0) __builtin_amdgcn_s_sleep(1);
      }
    };
    auto row8 = [&](int U) { return U < 8 ? (xj & 1) * 16 + U * 2 + e : q * 8 + (U - 8) * 2 + e; };
    auto fetch = [&](int U) {
      const int r8 = row8(U);
      const unsigned sld = (unsigned)((U < 8 ? b & ~1 : 256 + b - q) * SLAB + (r8 >> 4) * 32768 + ((r8 >> 1) & 7) * 2048 + (r8 & 1) * 128);
#pragma unroll
      for (int s = 0; s < 4; ++s)
        __builtin_amdgcn_raw_ptr_buffer_load_lds(rp, (LAS void*)(lds + RING_BYTES + e * 8192 + (s >> 1) * 4096 + (s & 1) * 1024), 16, vld, sld + (s & 1) * SLAB + (s >> 1) * 16384, 0, AUX_SYS);
      if (U >= 8) {
#pragma unroll
        for (int s = 0; s < 4; ++s)
          __builtin_amdgcn_raw_ptr_buffer_load_lds(rp, (LAS void*)(lds + RING_BYTES + e * 8192 + (s >> 1) * 4096 + 2048 + (s & 1) * 1024), 16, vld, sld + (2 + (s & 1)) * SLAB + (s >> 1) * 16384, 0, AUX_SYS);
      }
    };
    const int skip = SKIP(ltype, idx);
    bool have = false;
    if (DYN) {
      while (nb > NST + 1 - TS0) { STEP(); if (THR_N > 0 && NST - nb >= THR_S && NST - nb < THR_N) __builtin_amdgcn_s_sleep(THR_C); }
      const unsigned ta = (unsigned)__builtin_amdgcn_s_memrealtime(), ca = (unsigned)__builtin_amdgcn_s_memtime();
      while (nb > NST + 1 - TS) { STEP(); if (THR_N > 0 && NST - nb >= THR_S && NST - nb < THR_N) __builtin_amdgcn_s_sleep(THR_C); }
      const unsigned tb = (unsigned)__builtin_amdgcn_s_memrealtime(), cb = (unsigned)__builtin_amdgcn_s_memtime();
      if (idx == REP && e == 0 && lane == 0)
        __hip_atomic_store((unsigned long long*)(myws + WS_PACE + xj * 32), ((unsigned long long)call << 32) | (CLK ? (unsigned)((float)(tb - ta) * 172000.f / (float)(cb - ca)) : (tb << 16) | ((tb - ta) & 0xffffu)), __ATOMIC_RELAXED, __HIP_MEMORY_SCOPE_SYSTEM);
      STEP(); STEP();
      const unsigned long long t0 = __builtin_amdgcn_s_memrealtime();
      u32x2 v;
      for (;;) {
        asm volatile("global_load_dwordx2 %0, %1, off sc0 sc1" : "=v"(v) : "v"(myws + WS_PACE + ((xj & 4) + (lane & 3)) * 32) : "memory");
        if (nb > NST + 4 - MAILS) STEP(); else __builtin_amdgcn_s_sleep(1);
        asm volatile("s_waitcnt vmcnt(0)" : "+v"(v) :: "memory");
        if (__all(v.y == call)) break;
        if (__builtin_amdgcn_s_memrealtime() - t0 > DEADLINE) {
          if (lane == 0) __hip_atomic_store(myws + WS_TMO, call, __ATOMIC_RELAXED, __HIP_MEMORY_SCOPE_AGENT);
          bad = true; break;
        }
      }
      int f[4], ad[4], step = 0;
#pragma unroll
      for (int i = 0; i < 4; ++i) {
        const unsigned w = __builtin_amdgcn_readlane(v.x, i), w0 = __builtin_amdgcn_readlane(v.x, 0);
        const int dur = w & 0xffff, rel = (short)((w >> 16) - (w0 >> 16));
        f[i] = rel * (TS - TS0) + dur * (LAMB + ((i & 1) ? -BETA : BETA) + (xj < 4 && i < 2 ? FARB : 0)); ad[i] = 0; step += dur;
      }
      step >>= 1;
      for (int it = 0; it < 6; ++it) {
        if (nb > NST + 4 - MAILS) STEP();
        int hi = f[0], lo = f[0], ih = 0, il = 0;
#pragma unroll
        for (int i = 1; i < 4; ++i) { if (f[i] > hi) { hi = f[i]; ih = i; } if (f[i] < lo) { lo = f[i]; il = i; } }
        if (bad || (hi - lo) * 8 <= step * 9) break;
#pragma unroll
        for (int i = 0; i < 4; ++i) { if (i == ih) { f[i] -= step; ad[i] -= 2; } if (i == il) { f[i] += step; ad[i] += 2; } }
      }
      const int mine = q == 0 ? ad[0] : q == 1 ? ad[1] : q == 2 ? ad[2] : ad[3];
      const int before = (q > 0 ? ad[0] : 0) + (q > 1 ? ad[1] : 0) + (q > 2 ? ad[2] : 0);
      NST += mine; nb += mine;
      if (e == 0) {
        __hip_atomic_store(myws + WS_MAIL + b * 64 + lane, calcB(MAILS + 6 + ((lane - MAILS - 6) & 63), 32 + mine, q * 32 + before) | (unsigned)(32 + mine), __ATOMIC_RELAXED, __HIP_MEMORY_SCOPE_SYSTEM);
        if (nb > NST + 3 - MAILS) STEP();
        WAITV(0);
      }
    }
    while (nb > NST - (G0 < 64 ? 128 - G0 : G0) - ELAG) STEP();
    for (int U = 0; U < 12; ++U) {
      int len = 4;
      if (!have) {
        wait_gen(U == 0 ? myws + WS_DUMP + ((b & ~1) + (lane & 1)) * 32 : fl1);
        if (U == 0) STAMPAT(0); else STAMPAT(2);
        fetch(U);
        STEP();
        len = 4 - ((NST + 2 - nb + ((idx >> 2) & 3)) & 3);
        if (len < (U == 0 ? MINL0 : MINL1)) { for (int i = 0; i < len; ++i) STEP(); len = 4; }
        WAITV(0);
      } else {
        if ((skip >> U) & 1) { STEP(); STEP(); STEP(); STEP(); }
        WAITV(2);
      }
      u32x2 L[2][2][2]; u32x4 o[2];
      float y[2][8];
#define RD8(off) asm volatile("ds_read_b64 %0, %8\n\tds_read_b64 %1, %8 offset:128\n\tds_read_b64 %2, %8 offset:1024\n\tds_read_b64 %3, %8 offset:1152\n\t" \
                   "ds_read_b64 %4, %8 offset:64\n\tds_read_b64 %5, %8 offset:192\n\tds_read_b64 %6, %8 offset:1088\n\tds_read_b64 %7, %8 offset:1216\n\ts_waitcnt lgkmcnt(0)" \
                   : "=&v"(L[0][0][0]), "=&v"(L[0][0][1]), "=&v"(L[0][1][0]), "=&v"(L[0][1][1]), "=&v"(L[1][0][0]), "=&v"(L[1][0][1]), "=&v"(L[1][1][0]), "=&v"(L[1][1][1]) : "v"(vrd + (off)) : "memory")
      unsigned vf = 0;
      if (U == 7) asm volatile("global_load_dword %0, %1, off sc0 sc1" : "=v"(vf) : "v"(fl1) : "memory");
      RD8(0);
#pragma unroll
      for (int h = 0; h < 2; ++h)
#pragma unroll
        for (int d = 0; d < 4; ++d) {
          const float a0 = e_lo(L[h][0][d >> 1][d & 1]), a1 = e_hi(L[h][0][d >> 1][d & 1]), c0 = e_lo(L[h][1][d >> 1][d & 1]), c1 = e_hi(L[h][1][d >> 1][d & 1]);
          y[h][2 * d] = e_add(a0, c0); y[h][2 * d + 1] = e_add(a1, c1);
        }
      if (U >= 8) RD8(2048);
      if (len > 3) STEP();
      have = U < 11;
      if (U == 7) { asm volatile("s_waitcnt vmcnt(0)" : "+v"(vf) :: "memory"); have = __all(vf >= call); }
      if (U >= 8) {
#pragma unroll
        for (int h = 0; h < 2; ++h)
#pragma unroll
          for (int d = 0; d < 4; ++d)
#pragma unroll
            for (int s = 0; s < 2; ++s) {
              const float a0 = e_lo(L[h][s][d >> 1][d & 1]), a1 = e_hi(L[h][s][d >> 1][d & 1]);
              y[h][2 * d] = e_add(y[h][2 * d], a0); y[h][2 * d + 1] = e_add(y[h][2 * d + 1], a1);
            }
      }
      SCHED0();
      if (have && e == 0) fetch(U + 1);
      if (len > 2) STEP();
#pragma unroll
      for (int d = 0; d < 4; ++d) o[0][d] = e_gelu2(y[0][2 * d], y[0][2 * d + 1], k1v);
      SCHED0();
      if (have && e == 1) fetch(U + 1);
      STEP();
#pragma unroll
      for (int d = 0; d < 4; ++d) o[1][d] = e_gelu2(y[1][2 * d], y[1][2 * d + 1], k1v);
      if (bad) o[0] = o[1] = (u32x4){0x7fc07fc0u, 0x7fc07fc0u, 0x7fc07fc0u, 0x7fc07fc0u};
      const unsigned te = (tab >> (U < 8 ? 0 : 8)) & 0xff;
      auto ro = __builtin_amdgcn_make_buffer_rsrc(p.out[(te >> 3) & 7], 0, 8192 * NS_OUT * 2, 0x00020000);
      const unsigned so = (unsigned)((p.rank * MS + (te >> 6) * BM + row8(U) * 8) * (NS_OUT * 2) + (te & 7) * (BN * 2));
      __builtin_amdgcn_raw_buffer_store_b128(o[0], ro, vst, so, AUX_SYS);
      __builtin_amdgcn_raw_buffer_store_b128(o[1], ro, vst, so + 4 * (NS_OUT * 2), AUX_SYS);
      asm volatile("s_nop 1");
      STEP();
      if (U == 7) STAMPAT(1);
    }
    STAMPAT(3);
    STEP();
    WAITV(0);
    STAMPAT(4);
    const unsigned dl = (tab >> (8 * (lane & 3) + 3)) & 7;
    const int slot = (b * 10 + lane * 2 + e) * 32;
    if (lane < 2 && dl != (unsigned)p.rank) __hip_atomic_store(p.ws[dl] + WS_RECV + slot, call, __ATOMIC_RELAXED, __HIP_MEMORY_SCOPE_SYSTEM);
    while (nb > 0) STEP();
    STAMPAT(5);
    if (lane < 2) {
      if (dl != (unsigned)p.rank) {
        if (wait_flag(myws + WS_RECV + slot, call, myws + WS_TMO))
          __hip_atomic_store((unsigned short*)p.out[p.rank] + (size_t)(p.rank * MS) * NS_OUT, (unsigned short)0x7fc0, __ATOMIC_RELAXED, __HIP_MEMORY_SCOPE_SYSTEM);
      }
    }
    if (e == 0 && lane == 0) __hip_atomic_store(myws + WS_CNT + b * 32, call, __ATOMIC_RELAXED, __HIP_MEMORY_SCOPE_AGENT);
    STAMPAT(6);
    if (e == 0) STAMPOUT(180224);
  } else if (wid >= 4) {
    const int dw = wid - 4;
    unsigned vcall;
    asm volatile("global_load_dword %0, %1, off sc1" : "=v"(vcall) : "v"(myws + WS_CNT + b * 32) : "memory");
    constexpr int NA = 16, NB = 8;
    auto rx = __builtin_amdgcn_make_buffer_rsrc((char*)p.x - 4096, 0, MS * KD * 2 + 4096, 0x00020000);
    auto rw = __builtin_amdgcn_make_buffer_rsrc((void*)p.w, 0, KD * ND * 2, 0x00020000);
    unsigned vA[NA], vB[NB];
#pragma unroll
    for (int j = 0; j < NB; ++j) { vB[j] = (unsigned)((4 * (dw * NB + j) + (lane >> 4)) * (ND * 2) + (((lane & 15) ^ (((lane >> 4) | (((j >> 1) & 1) << 2)) << 1)) * 16) - (j & 3) * 1024); PIN(vB[j]); }
    unsigned vt = calcB(lane, 32, 0), vn;
    auto offB = [&](int t) -> unsigned { return __builtin_amdgcn_readlane(vt, t); };
    auto b2a = [&](unsigned sB) -> unsigned { return (sB >> 28) * (BM * KD * 2) + ((sB >> 20) & 127) * (BK * 2); };
#define Q4(r, l, v, i, s) do { __builtin_amdgcn_raw_ptr_buffer_load_lds(r, (LAS void*)(l), 16, v[i], s, 0, 0); __builtin_amdgcn_raw_ptr_buffer_load_lds(r, (LAS void*)(l), 16, v[i + 1], s, 1024, 0); \
  __builtin_amdgcn_raw_ptr_buffer_load_lds(r, (LAS void*)(l), 16, v[i + 2], s, 2048, 0); __builtin_amdgcn_raw_ptr_buffer_load_lds(r, (LAS void*)(l), 16, v[i + 3], s, 3072, 0); } while (0)
    auto issueB = [&](unsigned sB, unsigned lb) {
      sB &= 0x0fffff00u;
#pragma unroll
      for (int j = 0; j < NB; j += 4) Q4(rw, (size_t)(lb + A_BYTES + (dw * NB + j) * 1024), vB, j, sB);
    };
    auto issueA = [&](unsigned sA, unsigned lb) {
#pragma unroll
      for (int i = 0; i < NA; i += 4) Q4(rx, (size_t)(lb + (dw * NA + i) * 1024), vA, i, sA);
    };
    issueB(offB(0), ldsb + 2 * STAGE_BYTES);
    SCHED0();
#pragma unroll
    for (int i = 0; i < NA; ++i) { vA[i] = (unsigned)((8 * ((dw * NA + i) ^ (2 * cx)) + (lane >> 3)) * (KD * 2) + 4096 + (((lane & 7) ^ ((4 * (i & 1) + (lane >> 4)) & 7)) * 16) - (i & 3) * 1024); PIN(vA[i]); }
    issueA(b2a(offB(0)), ldsb);
    SCHED0();
    vn = calcB(64 + lane, 32, q * 32); PIN(vn);
    issueB(offB(1), ldsb); issueA(b2a(offB(1)), ldsb + STAGE_BYTES); issueB(offB(2), ldsb + STAGE_BYTES);
    asm volatile("s_waitcnt vmcnt(24)" : "+v"(vcall) :: "memory");
    BAR();
    vcall += 1;
    BAR();
    unsigned lb = ldsb + 2 * STAGE_BYTES, sB = offB(3), sA = b2a(offB(2));
    int i = 2;
    auto run = [&](int end) {
      do {
        issueA(sA, lb);
        SCHED0();
        sA = b2a(sB);
        unsigned cB = sB & 0x0fffff00u, cl = lb;
        sB = offB(i + 2);
        lb = lb == ldsb + 2 * STAGE_BYTES ? ldsb : lb + STAGE_BYTES;
        ++i;
        asm volatile("" : "+s"(sB), "+s"(sA), "+s"(cB), "+s"(cl), "+s"(lb), "+s"(i));
        SCHED0();
        issueB(cB, cl);
        WAITV(24); BAR();
      } while (i < end);
    };
    auto publish = [&](int slot) {
      if (dw == 0 && lane == 0) __hip_atomic_store(myws + WS_DUMP + slot * 32, vcall, __ATOMIC_RELAXED, __HIP_MEMORY_SCOPE_SYSTEM);
    };
    run(62);
    vt = vn;
    run(G0 + PUBD + 3);
    publish(b);
    run(MAILS + 3);
    if (DYN) asm volatile("global_load_dword %0, %1, off sc0 sc1" : "=v"(vn) : "v"(myws + WS_MAIL + b * 64 + lane) : "memory");
    else vn = calcB(MAILS + 6 + ((lane - MAILS - 6) & 63), 32, q * 32);
    run(MAILS + 4);
    PIN(vn);
    vt = vn;
    run(G1 + PUBD + 3);
    publish(256 + b);
    run(DYN ? G1 + (int)(RFL(vt) & 0xff) : NST);
    WAITV(0); BAR();
    BAR();
  } else {
    unsigned vcall;
    asm volatile("global_load_dword %0, %1, off sc1" : "=v"(vcall) : "v"(myws + WS_CNT + b * 32) : "memory");
    const int wm = wid >> 1, wn = wid & 1;
    unsigned aoff[2], boff[4];
    {
      const int l15 = lane & 15, g = lane >> 4;
#pragma unroll
      for (int ks = 0; ks < 2; ++ks) aoff[ks] = ldsb + (unsigned)((wm * 128 + l15) * 128 + (((ks * 4 + g) ^ (l15 >> 1)) * 16));
      const int qq = l15 >> 2, pp = lane & 3, h = qq | ((g & 1) << 2);
#pragma unroll
      for (int ni = 0; ni < 4; ++ni) boff[ni] = ldsb + (unsigned)(2 * STAGE_BYTES + A_BYTES + (8 * g + qq) * 256 + (((wn * 4 + ni) ^ h) * 32) + 8 * pp);
    }
    f32x4 acc[8][4];
    bf16x8 a[4][2]; bf16x4 bb[2][4][2][2];
    int bufn = 0; unsigned dlt;

#define RDA(slot, ks, mi) a[slot][ks] = *(LAS bf16x8*)(size_t)(aoff[ks] + (mi) * 2048)
#define RDB(P, ni, ks, t) bb[P][ni][ks][t] = __builtin_amdgcn_ds_read_tr16_b64_v4bf16((LAS bf16x4*)(size_t)(boff[ni] + (ks) * 8192 + (t) * 1024))
#define BF(P, ni, ks) __builtin_shufflevector(bb[P][ni][ks][0], bb[P][ni][ks][1], 0, 1, 2, 3, 4, 5, 6, 7)
#define MF(ZERO, P, mi, ni, ks) do { if (ZERO) asm volatile("v_mfma_f32_16x16x32_bf16 %0, %1, %2, 0" : "+v"(acc[mi][ni]) : "v"(BF(P, ni, ks)), "v"(a[(mi) & 3][ks])); \
   else asm volatile("v_mfma_f32_16x16x32_bf16 %0, %1, %2, %0" : "+v"(acc[mi][ni]) : "v"(BF(P, ni, ks)), "v"(a[(mi) & 3][ks])); } while (0)
#define CVT(k, i, T) asm volatile("v_cvt_pk_bf16_f32 %0, %1, %2" : "=v"(T[i]) : "v"(acc[(k) >> 1][2 * ((k) & 1) + ((i) >> 1)][2 * ((i) & 1)]), "v"(acc[(k) >> 1][2 * ((k) & 1) + ((i) >> 1)][2 * ((i) & 1) + 1]))
#define PUT(k, T) __builtin_amdgcn_raw_buffer_store_b128((u32x4){T[0], T[1], T[2], T[3]}, rd, vl + ((k) & 1) * 1024, sd + (unsigned)((((k) >> 1) ^ cx) * 2048), AUX_SYS)
#define STAGE(KIND, P, DRAIN) do { \
    _Pragma("unroll") for (int mi = 0; mi < 8; ++mi) { \
      _Pragma("unroll") for (int j = 0; j < 8; ++j) { \
        MF(KIND == 1 && (j & 1) == 0, P, mi, j >> 1, j & 1); SCHED0(); \
        if (j == (mi == 5 ? BARJ + 1 : 0)) RDA((mi + 3) & 3, 0, (mi + 3) & 7); \
        if (j == (mi == 4 ? 1 : 4)) RDA((mi + 3) & 3, 1, (mi + 3) & 7); \
        if (mi < 4 && (j == 1 || j == 2 || j == 5 || j == 6)) { const int r_ = (j < 4 ? j - 1 : j - 3); RDB((P) ^ 1, mi, r_ >> 1, r_ & 1); } \
        if (mi == 2 && j == 3) { bufn = bufn == 2 ? 0 : bufn + 1; asm volatile("" : "+s"(bufn)); } \
        if (mi == 3 && j == 3) { dlt = bufn == 0 ? (unsigned)(-2 * STAGE_BYTES) : (unsigned)STAGE_BYTES; asm volatile("" : "+s"(dlt)); } \
        if (mi == 4 && j == 2) { aoff[0] += dlt; PIN(aoff[0]); } \
        if (mi == 4 && j == 4) { aoff[1] += dlt; PIN(aoff[1]); } \
        if (mi == 4 && j == 6) { boff[0] += dlt; PIN(boff[0]); } \
        if (mi * 8 + j == 40 + BARJ - 1) { if (DRAIN) asm volatile("s_waitcnt vmcnt(0)" WL ::: "memory"); else if (LWAIT) asm volatile("s_waitcnt" WL ::: "memory"); } \
        if (mi * 8 + j == 40 + BARJ) BAR(); \
        if (mi == 5 && j == 6) { boff[1] += dlt; PIN(boff[1]); } \
        if (mi == 6 && j == 2) { boff[2] += dlt; PIN(boff[2]); } \
        if (mi == 6 && j == 6) { boff[3] += dlt; PIN(boff[3]); } \
        if (KIND == 2 && mi >= 1) { if (j < 4) CVT(mi - 1, j & 3, t0); if (j == 5) PUT(mi - 1, t0); } \
        if (KIND == 1) { \
          if (mi < 7) { if (j < 4) CVT(mi + 7, j & 3, t0); if (j == 5) PUT(mi + 7, t0); } \
          if (mi == 6 && j >= 4) CVT(14, j & 3, t1); \
          if (mi == 7) { if (j < 4) CVT(15, j & 3, t0); if (j == 1) PUT(14, t1); if (j == 6) PUT(15, t0); } \
        } \
        SCHED0(); \
      } \
    } } while (0)

    __builtin_amdgcn_s_setprio(2);
    STAMPAT(0);
    BAR();
    STAMPAT(1);
#pragma unroll
    for (int r = 0; r < 16; ++r) RDB(0, r >> 2, (r >> 1) & 1, r & 1);
#pragma unroll
    for (int mi = 0; mi < 3; ++mi) { RDA(mi, 0, mi); RDA(mi, 1, mi); }
#pragma unroll
    for (int ni = 0; ni < 4; ++ni) { boff[ni] -= 2 * STAGE_BYTES; PIN(boff[ni]); }
    asm volatile("s_waitcnt vmcnt(0)" : "+v"(vcall) :: "memory");
    const unsigned call = RFL(vcall) + 1;
    SCHED0();
    BAR();
    SCHED0();
    auto rp = __builtin_amdgcn_make_buffer_rsrc((char*)myws + PART_OFF, 0, PART_BYTES, 0x00020000);
    auto rd = __builtin_amdgcn_make_buffer_rsrc((char*)myws + PART_OFF, 0, 0, 0x00020000);
    unsigned vl = lane * 16; PIN(vl);
    unsigned sd = 0, t0[4], t1[4], vmail = 0;
#pragma unroll
    for (int mi = 0; mi < 8; ++mi)
#pragma unroll
      for (int ni = 0; ni < 4; ++ni) asm volatile("" : "=v"(acc[mi][ni]));
    for (int g = 0; g < 3; ++g) {
      STAGE(1, 0, false);
      asm volatile("s_waitcnt vmcnt(16)" : "+v"(vmail) :: "memory");
      const int n = g == 0 ? G0 : g == 1 ? G1 - G0 : DYN ? (int)(RFL(vmail) & 0xff) : 32;
      for (int s = 2; s < n; s += 2) {
        STAGE(0, 1, PUBD == 1);
        STAGE(0, 0, PUBD == 2);
      }
      asm volatile("global_load_dword %0, %1, %2 sc0 sc1" : "=v"(vmail) : "v"(0u), "s"(myws + WS_MAIL + b * 64) : "memory");
      rd = rp;
      sd = (unsigned)((g * 256 + (g < 2 ? b : vb)) * SLAB + wid * 16384);
      STAGE(2, 1, false);
    }
    asm volatile("s_nop 7");
    __builtin_amdgcn_s_setprio(0);
    STAMPAT(2);

    const int ln = __builtin_amdgcn_mbcnt_hi(-1, __builtin_amdgcn_mbcnt_lo(-1, 0));
    const int v0 = vb - q;
#pragma unroll
    for (int k = 7; k < 12; ++k) {
      unsigned t[4];
#pragma unroll
      for (int i = 0; i < 4; ++i) CVT(k, i, t);
      PUT(k, t); asm volatile("s_nop 1" :: "v"(t[0]), "v"(t[1]), "v"(t[2]), "v"(t[3]));
    }
    asm volatile("s_waitcnt vmcnt(0)" : "+v"(vmail) :: "memory");
    STAMPAT(3);
    bool bad = false;
    if (ln == 0) __hip_atomic_store(myws + WS_DUMP3 + (vb * 4 + wid) * 32, call, __ATOMIC_RELAXED, __HIP_MEMORY_SCOPE_SYSTEM);
    if (ln < 4 && ln != q) bad = wait_flag(myws + WS_DUMP3 + ((v0 + ln) * 4 + wid) * 32, call, myws + WS_TMO);
    bad = __any(bad);
    asm volatile("" ::: "memory");
    STAMPAT(4);
    f32x4 own[2][4];
    {
      u32x4 L[3][2][2];
#pragma unroll
      for (int s = 0; s < 3; ++s) {
        const int pq = (q + 1 + s) & 3;
        const unsigned sl = (unsigned)((2 * 256 + v0 + pq) * SLAB + wid * 16384 + q * 4096);
#pragma unroll
        for (int m2 = 0; m2 < 2; ++m2)
#pragma unroll
          for (int h = 0; h < 2; ++h) L[s][m2][h] = __builtin_amdgcn_raw_buffer_load_b128(rp, ln * 16 + h * 1024, sl + m2 * 2048, AUX_SYS);
      }
#pragma unroll
      for (int m2 = 0; m2 < 2; ++m2)
#pragma unroll
        for (int ni = 0; ni < 4; ++ni) {
          own[m2][ni] = acc[6 + m2][ni];
#pragma unroll
          for (int s = 0; s < 3; ++s) {
            const unsigned u0 = L[s][m2][ni >> 1][(ni & 1) * 2], u1 = L[s][m2][ni >> 1][(ni & 1) * 2 + 1];
            own[m2][ni] += (f32x4){__builtin_bit_cast(float, u0 << 16), __builtin_bit_cast(float, u0 & 0xffff0000u), __builtin_bit_cast(float, u1 << 16), __builtin_bit_cast(float, u1 & 0xffff0000u)};
          }
        }
    }
    const unsigned te = (tab >> 16) & 0xff;
    const int dest = (te >> 3) & 7;
    {
      LAS char* st = lds + wid * 4096;
      const int l15 = ln & 15, g = ln >> 4;
      const unsigned wo = (unsigned)(l15 * 128 + (g & 1) * 8);
      auto ro = __builtin_amdgcn_make_buffer_rsrc(p.out[dest], 0, 8192 * NS_OUT * 2, 0x00020000);
      const unsigned vo = (unsigned)((ln >> 3) * (NS_OUT * 2) + (((ln & 7) ^ (ln >> 3)) * 16));
      const unsigned so = (unsigned)((p.rank * MS + (te >> 6) * BM + wm * 128 + q * 32) * (NS_OUT * 2) + (te & 7) * (BN * 2) + wn * 128);
      u32x4 r[4];
#pragma unroll
      for (int m2 = 0; m2 < 2; ++m2) {
#pragma unroll
        for (int ni = 0; ni < 4; ++ni) {
          f32x4 v = own[m2][ni];
          u32x2 o; o.x = gelu_pk((f32x2){v[0], v[1]}); o.y = gelu_pk((f32x2){v[2], v[3]});
          if (bad) o = (u32x2){0x7fc07fc0u, 0x7fc07fc0u};
          *(LAS u32x2*)(st + m2 * 2048 + wo + (((ni * 2 + (g >> 1)) ^ (l15 & 7)) * 16)) = o;
        }
        __builtin_amdgcn_fence(__ATOMIC_RELEASE, "wavefront"); __builtin_amdgcn_wave_barrier(); __builtin_amdgcn_fence(__ATOMIC_ACQUIRE, "wavefront");
        r[2 * m2] = *(LAS u32x4*)(st + m2 * 2048 + ln * 16); r[2 * m2 + 1] = *(LAS u32x4*)(st + m2 * 2048 + 1024 + ln * 16);
        if (m2 == 0) { __builtin_amdgcn_raw_buffer_store_b128(r[0], ro, vo, so, AUX_SYS); SCHED0(); asm volatile("s_nop 1"); SCHED0(); }
      }
#pragma unroll
      for (int j = 1; j < 4; ++j) {
        if (dest != p.rank) WAITV(0);
        __builtin_amdgcn_raw_buffer_store_b128(r[j], ro, vo, so + j * 8 * (NS_OUT * 2), AUX_SYS);
      }
    }
    STAMPAT(5);
    WAITV(0);
    STAMPAT(6);
    if (ln == 0 && dest != p.rank) {
      const int slot = (b * 10 + 6 + wid) * 32;
      __hip_atomic_store(p.ws[dest] + WS_RECV + slot, call, __ATOMIC_RELAXED, __HIP_MEMORY_SCOPE_SYSTEM);
      if (wait_flag(myws + WS_RECV + slot, call, myws + WS_TMO))
        __hip_atomic_store((unsigned short*)p.out[p.rank] + (size_t)(p.rank * MS) * NS_OUT, (unsigned short)0x7fc0, __ATOMIC_RELAXED, __HIP_MEMORY_SCOPE_SYSTEM);
    }
    STAMPAT(7);
    if (wid == 0) STAMPOUT(163840);
  }
}

extern "C" void kernel_launch(int rank, int nranks, ncclComm_t comm, hipStream_t stream,
                              void* const* d_in, const int* in_sizes, int n_in,
                              void* d_out, int out_size, const int* in_dtypes, int out_dtype,
                              void* const* d_out_all, void* const* d_ws_all, size_t ws_bytes) {
  if (nranks != NR || (unsigned)rank >= (unsigned)NR || ws_bytes < PART_OFF + (size_t)PART_BYTES) return;
  Params p;
  p.x = d_in[0]; p.w = d_in[1]; p.rank = rank;
  for (int i = 0; i < NR; ++i) { p.out[i] = d_out_all[i]; p.ws[i] = (unsigned*)d_ws_all[i]; }
  static const int label[8] = {0, 4, 1, 2, 5, 6, 3, 7};
  int peer[8]; for (int r = 0; r < 8; ++r) peer[label[rank] ^ label[r]] = r;
  unsigned char lst[8][32], phs[8][32]; int cnt[8] = {0}, next[8] = {0}, nextb[8] = {0}, nq = 0;
  auto add = [&](int grp, int delta, int tn, int tm, int ph) { phs[grp][cnt[grp]] = (unsigned char)ph; lst[grp][cnt[grp]++] = (unsigned char)(peer[delta] * 8 + tn + tm * 64); };
#define X 16
  static const unsigned char dl0[64] = {9,9,9,1,2,3,1,2, 3,1,2,3,1,2,3,1, 4,5,6,7,4,5,6,7, 4,5,6,7,4,5,6,7,
                                        1+X,2+X,3+X,2,1+X,2+X,3+X,3, 4+X,5+X,6+X,7+X,4+X,5+X,6+X,7+X,
                                        1,2,3,9,0,0,0,0+X, 4,5,6,7,0,0,0,0+X};
  static const unsigned char dl1[64] = {9,9,1,2,3,1,2,3, 1,2,3,1,2,3,1,2, 4,5,6,7,4,5,6,7, 4,5,6,7,4,5,6,7,
                                        1+X,2+X,3+X,9,1+X,2+X,3+X,3, 4+X,5+X,6+X,7+X,4+X,5+X,6+X,7+X,
                                        1,2,3,9,0,0,0,0+X, 4,5,6,7,0,0,0,0+X};
  static const unsigned char dl2[64] = {9,9,1+X,2,3+X,1,2+X,3, 1+X,2+X,3+X,1,2,3,1,2, 4+X,5+X,6+X,7,4,5,6,7+X, 4,5,6,7+X,4+X,5+X,6+X,7,
                                        1,2,3,9,1,2,3,3, 4,5,6,7,4,5,6,7,
                                        1,2,3,9,0,0,0,0+X, 4,5,6,7,0,0,0+X,0};
  static const unsigned char dl3[64] = {9,9,1,2,3,1,2,3, 1,2,3,1,2,3,1,2, 4,5,6,7,4,5,6,7+X, 4,5,6,7+X,4,5,6,7,
                                        1+X,2+X,3+X,9,1+X,2+X,3+X,3, 4+X,5+X,6+X,7,4+X,5+X,6+X,7,
                                        1,2,3,9,0,0,0,0+X, 4,5,6,7,0,0,0,0+X};
#undef X
  const unsigned char* dl = PLAN == 0 ? dl0 : PLAN == 1 ? dl1 : PLAN == 2 ? dl2 : dl3;
  static const unsigned char good[6] = {0, 2, 3, 4, 6, 7};
  for (int i = 0; i < 64; ++i) {
    const int d0 = dl[i] & 15;
    if (d0 == 9) {
      for (int tm = 0; tm < 4; ++tm) add(i >> 3, 4 + ((tm + nq) & 3), 7, tm, (tm + nq) & 3);
      ++nq;
    } else {
      const int tn = (dl[i] & 16) ? 1 + 4 * nextb[d0]++ : good[next[d0]++];
      for (int tm = 0; tm < 4; ++tm) add(i >> 3, d0, tn, tm, i & 3);
    }
  }
  for (int b = 0; b < 256; ++b) {
    const int xj = b & 7, i = b >> 3, vi = i ^ ((xj & 1) << 4), h = xj >> 2;
    p.tab[b] = lst[xj >> 1][i] | (lst[4 + h][i] << 8) | (lst[6 + h][vi] << 16) | ((unsigned)phs[xj >> 1][i] << 24);
  }
  gemm_a2a<<<256, 512, 0, stream>>>(p);
}
```
